```python
import numpy as np

kernel_name = "gpu_ag_gemm_m2048_k2048_n2048_f32_gelu"


ABSMAX_RTOL = 0.001

COMPUTE_DTYPE = "float32"
EPILOGUE = "gelu"
IN_QUANT = None
OUT_QUANT = None
IN_DTYPES = {"x": "float32", "w_mat": "float32"}
WORLD_SIZE = 8

def setup_inputs():

    x = np.random.randn(2048, 2048).astype(np.float32)
    w_mat = (np.random.randn(2048, 2048) / np.sqrt(2048)).astype(np.float32)
    return {"x": x, "w_mat": w_mat}

def _epilogue(y):
    c = 0.7978845608028654
    return (0.5 * y * (1.0 + np.tanh(c * (y + 0.044715 * y ** 3)))).astype(np.float32)

def reference(x, w_mat):
    y = (x @ w_mat).astype(np.float32)
    return _epilogue(y)
```

```cpp
#include <hip/hip_runtime.h>
#include <rccl/rccl.h>
#include <cstdio>

#define HIP_CHECK(expr) do { hipError_t _e = (expr); if (_e != hipSuccess) { \
    fprintf(stderr, "HIP error %s at %s:%d\n", hipGetErrorString(_e), __FILE__, __LINE__); \
    return; } } while (0)

typedef unsigned u32x4 __attribute__((ext_vector_type(4)));
typedef float f32x4 __attribute__((ext_vector_type(4)));
typedef float f32x2 __attribute__((ext_vector_type(2)));
typedef float f32x16 __attribute__((ext_vector_type(16)));
typedef _Float16 f16x8 __attribute__((ext_vector_type(8)));

constexpr int NR = 8;
constexpr int KDIM = 2048, NLOC = 256, MLOC = 256;
constexpr int NWG = 256, NTHR = 512;
constexpr size_t OFF_CNT = 0;
constexpr size_t OFF_TMO = 32768;
constexpr size_t OFF_FLAG = 65536;
constexpr size_t OFF_DATA = 1u << 20;
constexpr size_t SLOT_BYTES = 8u << 20;
constexpr size_t WS_NEED = OFF_DATA + 2 * SLOT_BYTES;
constexpr unsigned long long DEADLINE_TICKS = 50000000ull;

struct Args {
    const float* x;
    const float* w;
    float* out;
    char* ws[NR];
    int rank;
    int pad;
};

__device__ __forceinline__ void store_wt(void* p, u32x4 v) {
    asm volatile("global_store_dwordx4 %0, %1, off sc0 sc1\n\ts_nop 1" :: "v"(p), "v"(v) : "memory");
}

__device__ __forceinline__ float gelu(float y) {
    const float c = 0.7978845608028654f;
    float u = c * (y + 0.044715f * y * y * y);
    float e = __expf(2.0f * u);
    float t = 1.0f - 2.0f / (e + 1.0f);
    return 0.5f * y * (1.0f + t);
}

__global__ __launch_bounds__(NTHR, 2) void ag_gemm(Args a) {
    __shared__ float red[8 * 1024];
    __shared__ unsigned s_call;
    __shared__ unsigned s_bad;
    const int tid = threadIdx.x;
    const int lane = tid & 63;
    const int wv = __builtin_amdgcn_readfirstlane(tid >> 6);
    const int b = blockIdx.x;
    char* myws = a.ws[a.rank];
    if (tid == 0) {
        unsigned t = __hip_atomic_fetch_add((unsigned*)(myws + OFF_CNT + (size_t)b * 128), 1u,
                                            __ATOMIC_RELAXED, __HIP_MEMORY_SCOPE_AGENT);
        s_call = t + 1;
        s_bad = 0;
    }
    const int fi = wv & 3;
    const int prb = 2 * fi + (b & 1);
    const int pkf = b >> 1;
    const float* src = a.x + (size_t)(prb * 32 + (lane & 31)) * KDIM + pkf * 16 + (lane >> 5) * 8;
    f32x4 v0 = *(const f32x4*)src;
    f32x4 v1 = *(const f32x4*)(src + 4);
    __syncthreads();
    const unsigned call = s_call;
    const size_t slot = OFF_DATA + (size_t)(call & 1) * SLOT_BYTES;
    {
        f16x8 h;
        for (int j = 0; j < 4; ++j) { h[j] = (_Float16)v0[j]; h[4 + j] = (_Float16)v1[j]; }
        u32x4 hv = __builtin_bit_cast(u32x4, h);
        const size_t off = slot + ((size_t)(a.rank * 8 + prb) * 128 + pkf) * 1024 + lane * 16;
        for (int i = 0; i < 4; ++i) {
            int p = (a.rank + 1 + (wv >> 2) * 4 + i) & 7;
            store_wt(a.ws[p] + off, hv);
        }
        asm volatile("s_waitcnt vmcnt(0)" ::: "memory");
        if (lane < 4) {
            int p = (a.rank + 1 + (wv >> 2) * 4 + lane) & 7;
            char* base = a.ws[0];
            for (int q = 1; q < NR; ++q) if (p == q) base = a.ws[q];
            unsigned* f = (unsigned*)(base + OFF_FLAG) + ((a.rank * 8 + prb) * 128 + pkf);
            __hip_atomic_store(f, call, __ATOMIC_RELAXED, __HIP_MEMORY_SCOPE_SYSTEM);
        }
    }
    const int srank = b & 7;
    const int cg = (b >> 3) & 7;
    const int rl = b >> 6;
    f16x8 bf[16];
    {
        const float* wp = a.w + (size_t)(wv * 256 + (lane >> 5) * 8) * NLOC + cg * 32 + (lane & 31);
#pragma unroll
        for (int f = 0; f < 16; ++f) {
#pragma unroll
            for (int j = 0; j < 8; ++j) bf[f][j] = (_Float16)wp[(size_t)(f * 16 + j) * NLOC];
        }
    }
    unsigned* tmo = (unsigned*)(myws + OFF_TMO);
    for (int it = 0; it < 2; ++it) {
        const int rb = rl + 4 * it;
        {
            const unsigned* f = (const unsigned*)(myws + OFF_FLAG) + ((srank * 8 + rb) * 128 + wv * 16 + (lane & 15));
            unsigned long long t0 = __builtin_amdgcn_s_memrealtime();
            while (true) {
                unsigned v = __hip_atomic_load(f, __ATOMIC_RELAXED, __HIP_MEMORY_SCOPE_SYSTEM);
                if (__all((int)(v - call) >= 0)) break;
                if (__hip_atomic_load(tmo, __ATOMIC_RELAXED, __HIP_MEMORY_SCOPE_AGENT) == call ||
                    __builtin_amdgcn_s_memrealtime() - t0 > DEADLINE_TICKS) {
                    __hip_atomic_store(tmo, call, __ATOMIC_RELAXED, __HIP_MEMORY_SCOPE_AGENT);
                    s_bad = 1;
                    break;
                }
                __builtin_amdgcn_s_sleep(2);
            }
        }
        const char* ap = myws + slot + ((size_t)(srank * 8 + rb) * 128 + wv * 16) * 1024 + lane * 16;
        u32x4 af[16];
        asm volatile(
            "global_load_dwordx4 %0, %16, off sc0 sc1\n\t"
            "global_load_dwordx4 %1, %16, off offset:1024 sc0 sc1\n\t"
            "global_load_dwordx4 %2, %16, off offset:2048 sc0 sc1\n\t"
            "global_load_dwordx4 %3, %16, off offset:3072 sc0 sc1\n\t"
            "global_load_dwordx4 %4, %17, off sc0 sc1\n\t"
            "global_load_dwordx4 %5, %17, off offset:1024 sc0 sc1\n\t"
            "global_load_dwordx4 %6, %17, off offset:2048 sc0 sc1\n\t"
            "global_load_dwordx4 %7, %17, off offset:3072 sc0 sc1\n\t"
            "global_load_dwordx4 %8, %18, off sc0 sc1\n\t"
            "global_load_dwordx4 %9, %18, off offset:1024 sc0 sc1\n\t"
            "global_load_dwordx4 %10, %18, off offset:2048 sc0 sc1\n\t"
            "global_load_dwordx4 %11, %18, off offset:3072 sc0 sc1\n\t"
            "global_load_dwordx4 %12, %19, off sc0 sc1\n\t"
            "global_load_dwordx4 %13, %19, off offset:1024 sc0 sc1\n\t"
            "global_load_dwordx4 %14, %19, off offset:2048 sc0 sc1\n\t"
            "global_load_dwordx4 %15, %19, off offset:3072 sc0 sc1\n\t"
            "s_waitcnt vmcnt(0)"
            : "=&v"(af[0]), "=&v"(af[1]), "=&v"(af[2]), "=&v"(af[3]), "=&v"(af[4]), "=&v"(af[5]),
              "=&v"(af[6]), "=&v"(af[7]), "=&v"(af[8]), "=&v"(af[9]), "=&v"(af[10]), "=&v"(af[11]),
              "=&v"(af[12]), "=&v"(af[13]), "=&v"(af[14]), "=&v"(af[15])
            : "v"(ap), "v"(ap + 4096), "v"(ap + 8192), "v"(ap + 12288) : "memory");
        f32x16 acc = {};
#pragma unroll
        for (int f = 0; f < 16; ++f)
            acc = __builtin_amdgcn_mfma_f32_32x32x16_f16(__builtin_bit_cast(f16x8, af[f]), bf[f], acc, 0, 0, 0);
        if (it) __syncthreads();
#pragma unroll
        for (int r = 0; r < 16; ++r) {
            int row = (r & 3) + 8 * (r >> 2) + 4 * (lane >> 5);
            red[wv * 1024 + row * 32 + (lane & 31)] = acc[r];
        }
        __syncthreads();
        f32x2 s = {0.f, 0.f};
#pragma unroll
        for (int q = 0; q < 8; ++q) s += *(const f32x2*)&red[q * 1024 + tid * 2];
        bool bad = s_bad || __hip_atomic_load(tmo, __ATOMIC_RELAXED, __HIP_MEMORY_SCOPE_AGENT) == call;
        f32x2 o = {gelu(s[0]), gelu(s[1])};
        if (bad) { o[0] = __builtin_nanf(""); o[1] = o[0]; }
        const int row = tid >> 4, col = (tid & 15) * 2;
        *(f32x2*)(a.out + (size_t)(srank * 256 + rb * 32 + row) * NLOC + cg * 32 + col) = o;
    }
}

extern "C" void kernel_launch(int rank, int nranks, ncclComm_t comm, hipStream_t stream,
                              void* const* d_in, const int* in_sizes, int n_in,
                              void* d_out, int out_size, const int* in_dtypes, int out_dtype,
                              void* const* d_out_all, void* const* d_ws_all, size_t ws_bytes) {
    if (nranks != NR || ws_bytes < WS_NEED) { fprintf(stderr, "bad config\n"); return; }
    Args a;
    a.x = (const float*)d_in[0];
    a.w = (const float*)d_in[1];
    a.out = (float*)d_out;
    for (int p = 0; p < NR; ++p) a.ws[p] = (char*)d_ws_all[p];
    a.rank = rank;
    a.pad = 0;
    ag_gemm<<<NWG, NTHR, 0, stream>>>(a);
    HIP_CHECK(hipGetLastError());
}
```

```cpp
#include <hip/hip_runtime.h>
#include <rccl/rccl.h>
#include <cstdio>
#include <algorithm>
#include <vector>

#define HIP_CHECK(expr) do { hipError_t _e = (expr); if (_e != hipSuccess) { \
    fprintf(stderr, "HIP error %s at %s:%d\n", hipGetErrorString(_e), __FILE__, __LINE__); \
    return; } } while (0)

typedef unsigned u32x4 __attribute__((ext_vector_type(4)));
typedef unsigned u32x2 __attribute__((ext_vector_type(2)));
typedef float f32x4 __attribute__((ext_vector_type(4)));
typedef float f32x2 __attribute__((ext_vector_type(2)));
typedef _Float16 f16x8 __attribute__((ext_vector_type(8)));
typedef _Float16 f16x2 __attribute__((ext_vector_type(2)));

#define NX 11
#define NS_X 193
#define NS_O 193
#define NS_OUT 175
#define T_BASE 60
#define LEAD 8
#define POLL_SLP 1
#define BPOLL_SLP 2
#define VPOLL_SLP 2
#define CLR_START 400
#define CLR_STEP 9
#define Y_START 300
#define OUT_POS (256 + NY * 64)
#define TB_GAP 60
#define ROT 6
#define XMAP 1
constexpr int NR = 8;
constexpr int NY = 16 - NX;
constexpr int KDIM = 2048, NLOC = 256;
constexpr int NWG = 256, NTHR = 512;
constexpr size_t OFF_CNT = 0;
constexpr size_t OFF_TMO = 32768;
constexpr size_t OFF_TS = 33024;
constexpr size_t OFF_DIAG = 65536;
constexpr size_t OFF_DATA = 1u << 20;
constexpr size_t A_BYTES = 64u * 65536u;
constexpr size_t B_BYTES = 8u * 262144u;
constexpr size_t SLOT_BYTES = A_BYTES + B_BYTES;
constexpr int NSLOT = 3;
constexpr size_t WS_NEED = OFF_DATA + NSLOT * SLOT_BYTES;
constexpr unsigned DEADLINE_TICKS = 50000000u;
constexpr float WSCALE_F = 64.0f;
constexpr int PPITCH = 68;
constexpr unsigned HOLE = 0xffffffffu;

struct Args {
    const float* x;
    const float* w;
    char* self;
    float* out;
    int idx;
    int pad;
    char* link[8];
    float* outp[8];
    unsigned items[64];
    unsigned tiles[64];
};

__device__ __forceinline__ void store_wt(char* sbase, unsigned voff, u32x4 v) {
    asm volatile("global_store_dwordx4 %0, %1, %2 sc0 sc1\n\ts_nop 1" :: "v"(voff), "v"(v), "s"(sbase) : "memory");
}
__device__ __forceinline__ void store_wt2(char* sbase, unsigned voff, u32x2 v) {
    asm volatile("global_store_dwordx2 %0, %1, %2 sc0 sc1\n\ts_nop 1" :: "v"(voff), "v"(v), "s"(sbase) : "memory");
}
__device__ __forceinline__ unsigned pack2(float a, float b) {
    f16x2 h = {(_Float16)a, (_Float16)b};
    unsigned u = __builtin_bit_cast(unsigned, h);
    return u == 0u ? 0x80000000u : u;
}
__device__ __forceinline__ u32x4 pack8(f32x4 a, f32x4 b) {
    u32x4 r = {pack2(a[0], a[1]), pack2(a[2], a[3]), pack2(b[0], b[1]), pack2(b[2], b[3])};
    return r;
}
__device__ __forceinline__ unsigned gelu_bits(float y) {
    const float k0 = -2.0f * 0.7978845608028654f * 1.4426950408889634f;
    const float k1 = k0 * 0.044715f;
    float e = __builtin_amdgcn_exp2f(y * (k0 + k1 * y * y));
    unsigned u = __builtin_bit_cast(unsigned, y * __builtin_amdgcn_rcpf(1.0f + e));
    return u == HOLE ? 0x7fc00000u : u;
}
__device__ __forceinline__ unsigned long long now() { return __builtin_amdgcn_s_memrealtime(); }
__device__ __forceinline__ unsigned now32() { return (unsigned)__builtin_amdgcn_s_memrealtime(); }
__device__ __forceinline__ bool give_up(unsigned& spins, unsigned t0, unsigned* tmo, unsigned call) {
    if (__builtin_expect((++spins & 63u) != 0u, 1)) return false;
    unsigned t, u;
    asm volatile("v_mov_b32 %0, 0\n\tglobal_load_dword %0, %0, %1 sc1\n\ts_waitcnt vmcnt(0)" : "=&v"(t) : "s"(tmo) : "memory");
    if (__builtin_amdgcn_readfirstlane(t) == call || now32() - t0 > DEADLINE_TICKS) {
        asm volatile("v_mov_b32 %0, 0\n\tv_mov_b32 %1, %3\n\tglobal_store_dword %0, %1, %2 sc1\n\ts_nop 1"
                     : "=&v"(t), "=&v"(u) : "s"(tmo), "s"(call) : "memory");
        return true;
    }
    return false;
}
__device__ __forceinline__ void wait_until(unsigned t) {
    while ((int)(now32() - t) < 0) __builtin_amdgcn_s_sleep(1);
}
template <int NS> __device__ __forceinline__ unsigned pos_time(unsigned tstart, int pos) { return tstart + T_BASE + (unsigned)(pos * NS) / 100u; }

__device__ __forceinline__ void issue_ts(unsigned& v, const char* self, int lane) {
    asm volatile("global_load_dword %0, %1, %2 sc0 sc1" : "=&v"(v) : "v"((unsigned)(lane & 7) * 4u), "s"(self + OFF_TS) : "memory");
}
__device__ __forceinline__ unsigned earliest(unsigned v, unsigned tstart) {
    unsigned d = tstart - v;
    d = d < 100u ? d : 0u;
    unsigned m = 0;
#pragma unroll
    for (int i = 0; i < 8; ++i) m = max(m, (unsigned)__builtin_amdgcn_readlane(d, i));
    return tstart - m;
}
__device__ __forceinline__ void issue_x(f32x4& p, f32x4& q, const float* sb, unsigned voff) {
    asm volatile("global_load_dwordx4 %0, %2, %3\n\tglobal_load_dwordx4 %1, %2, %3 offset:16"
                 : "=&v"(p), "=&v"(q) : "v"(voff), "s"(sb) : "memory");
}
template <int N> __device__ __forceinline__ void wait2(f32x4& p, f32x4& q) {
    asm volatile("s_waitcnt vmcnt(%2)" : "+v"(p), "+v"(q) : "n"(N) : "memory");
}

#define LD1(i, off) "s_bitcmp1_b32 %[pend], " #i "\n\ts_cbranch_scc0 Lx" #i "_%=\n\tglobal_load_dwordx4 %" #i ", %[va], %[sb] offset:" #off " sc0 sc1\nLx" #i "_%=:\n\t"
__device__ __forceinline__ void load8(u32x4* r, unsigned pend, const char* sb, unsigned va) {
    asm volatile("s_nop 4\n\t"
                 LD1(0, -4096) LD1(1, -3072) LD1(2, -2048) LD1(3, -1024) LD1(4, 0) LD1(5, 1024) LD1(6, 2048) LD1(7, 3072)
                 "s_waitcnt vmcnt(0)"
                 : "+v"(r[0]), "+v"(r[1]), "+v"(r[2]), "+v"(r[3]), "+v"(r[4]), "+v"(r[5]), "+v"(r[6]), "+v"(r[7])
                 : [pend] "s"(pend), [va] "v"(va), [sb] "s"(sb)
                 : "memory", "scc");
}
__device__ __forceinline__ void issue8(u32x4* r, const char* sb, unsigned va) {
    asm volatile("s_nop 4\n\t"
                 "global_load_dwordx4 %0, %[va], %[sb] offset:-4096 sc0 sc1\n\t"
                 "global_load_dwordx4 %1, %[va], %[sb] offset:-3072 sc0 sc1\n\t"
                 "global_load_dwordx4 %2, %[va], %[sb] offset:-2048 sc0 sc1\n\t"
                 "global_load_dwordx4 %3, %[va], %[sb] offset:-1024 sc0 sc1\n\t"
                 "global_load_dwordx4 %4, %[va], %[sb] offset:0 sc0 sc1\n\t"
                 "global_load_dwordx4 %5, %[va], %[sb] offset:1024 sc0 sc1\n\t"
                 "global_load_dwordx4 %6, %[va], %[sb] offset:2048 sc0 sc1\n\t"
                 "global_load_dwordx4 %7, %[va], %[sb] offset:3072 sc0 sc1"
                 : "=&v"(r[0]), "=&v"(r[1]), "=&v"(r[2]), "=&v"(r[3]), "=&v"(r[4]), "=&v"(r[5]), "=&v"(r[6]), "=&v"(r[7])
                 : [va] "v"(va), [sb] "s"(sb)
                 : "memory");
}
template <int N> __device__ __forceinline__ void wait_frag(u32x4& r) {
    asm volatile("s_waitcnt vmcnt(%1)" : "+v"(r) : "n"(N) : "memory");
}
template <int N> __device__ __forceinline__ void wait8(u32x4* r) {
    asm volatile("s_waitcnt vmcnt(%8)"
                 : "+v"(r[0]), "+v"(r[1]), "+v"(r[2]), "+v"(r[3]), "+v"(r[4]), "+v"(r[5]), "+v"(r[6]), "+v"(r[7]) : "n"(N) : "memory");
}
__device__ __forceinline__ unsigned min4(u32x4 v) { return min(min(v[0], v[1]), min(v[2], v[3])); }
__device__ __forceinline__ unsigned max4(u32x4 v) { return max(max(v[0], v[1]), max(v[2], v[3])); }
__device__ __forceinline__ bool arrived(u32x4 v) { return __ballot(min4(v) == 0u) == 0ull; }
__device__ __forceinline__ bool arrived8(const u32x4* r) {
    unsigned m = min4(r[0]);
#pragma unroll
    for (int i = 1; i < 8; ++i) m = min(m, min4(r[i]));
    return __ballot(m == 0u) == 0ull;
}

template <int SLP, unsigned EMPTY>
__device__ __forceinline__ bool wait_words(const char* sb, unsigned vo, unsigned* tmo, unsigned call) {
    unsigned spins = 0;
    const unsigned t0 = now32();
    while (true) {
        unsigned v;
        asm volatile("global_load_dword %0, %1, %2 sc0 sc1\n\ts_waitcnt vmcnt(0)" : "=&v"(v) : "v"(vo), "s"(sb) : "memory");
        if (__ballot(v == EMPTY) == 0ull) return true;
        if (give_up(spins, t0, tmo, call)) return false;
        __builtin_amdgcn_s_sleep(SLP);
    }
}

#define STAMP(i) do {} while (0)

__global__ __launch_bounds__(NTHR, 2) void ag_gemm(Args a) {
    __shared__ __attribute__((aligned(16))) float part[2 * 8 * 16 * PPITCH];
    const int tid = threadIdx.x;
    const int lane = tid & 63;
    const int wv = __builtin_amdgcn_readfirstlane(tid >> 6);
    const int b = blockIdx.x;
    char* self = a.self;
    char* dst = a.link[wv];
    const float* xin = a.x;
    const float* win = a.w;
    char* outb = (char*)a.out;
    const int idx = a.idx;
    const unsigned tile = a.tiles[b & 63];
    asm volatile("" :: "s"(self), "s"(dst), "s"(xin), "s"(win), "s"(idx), "s"(outb), "s"(tile));
    const unsigned tstart = now32();
    unsigned* cnt = (unsigned*)(self + OFF_CNT + b * 128 + wv * 4);
    unsigned* tmo = (unsigned*)(self + OFF_TMO);

    if (__builtin_expect(b < 64, 1)) {
        const int kf = b;
        const unsigned xo = (unsigned)(((lane & 15) * KDIM + kf * 32 + (lane >> 4) * 8) * 4);
        constexpr size_t XB = 16 * KDIM;
        unsigned prev;
        if (wv < 3) {
            f32x4 p[NX], q[NX];
            issue_x(p[0], q[0], xin, xo);
            asm volatile("v_mov_b32 %0, 0\n\tglobal_load_dword %0, %0, %1 sc1" : "=&v"(prev) : "s"(cnt) : "memory");
            asm volatile("s_waitcnt vmcnt(0)" : "+v"(p[0]), "+v"(q[0]), "+v"(prev) :: "memory");
            unsigned tsv;
            issue_ts(tsv, self, lane);
#pragma unroll
            for (int i = 1; i < NX; ++i) issue_x(p[i], q[i], xin + i * XB, xo);
            const unsigned call = __builtin_amdgcn_readfirstlane(prev) + 1u;
            const size_t so = OFF_DATA + (call % NSLOT) * SLOT_BYTES + ((size_t)(idx * NX) * 64 + kf) * 1024;
            char* sb = dst + so;
            u32x4 h[NX];
            h[0] = pack8(p[0], q[0]);
            asm volatile("" : "+v"(h[0]));
            STAMP(0);
            wait_until(pos_time<NS_X>(tstart, kf));
            STAMP(1);
            store_wt(sb, lane * 16, h[0]);
            asm volatile("s_waitcnt vmcnt(%3)" : "+v"(p[1]), "+v"(q[1]), "+v"(tsv) : "n"(2 * (NX - 2) + 1) : "memory");
            const unsigned tref = earliest(tsv, tstart);
            h[1] = pack8(p[1], q[1]);
            asm volatile("" : "+v"(h[1]));
            STAMP(3);
            wait_until(pos_time<NS_X>(tref, 64 + kf));
            store_wt(sb + 65536, lane * 16, h[1]);
#pragma unroll
            for (int i = 2; i < NX; ++i) wait2<2>(p[i], q[i]);
#pragma unroll
            for (int i = 2; i < NX; ++i) { h[i] = pack8(p[i], q[i]); asm volatile("" : "+v"(h[i])); }
            STAMP(2);
#pragma unroll
            for (int i = 0; i < NX; ++i)
                if (i % 3 == wv) store_wt(self + so + (size_t)i * 65536, lane * 16, h[i]);
#pragma unroll
            for (int i = 2; i < NX; ++i) {
                wait_until(pos_time<NS_X>(tref, i * 64 + kf));
                if (i == NX - 1) STAMP(6);
                store_wt(sb + (size_t)i * 65536, lane * 16, h[i]);
            }
            __hip_atomic_store(cnt, call, __ATOMIC_RELAXED, __HIP_MEMORY_SCOPE_AGENT);
        } else if (wv < 7) {
            const int qd = wv - 3;
            f32x4 p[NY], q[NY];
            issue_x(p[0], q[0], xin + NX * XB, xo);
            asm volatile("v_mov_b32 %0, 0\n\tglobal_load_dword %0, %0, %1 sc1" : "=&v"(prev) : "s"(cnt) : "memory");
            asm volatile("s_waitcnt vmcnt(0)" : "+v"(p[0]), "+v"(q[0]), "+v"(prev) :: "memory");
            const unsigned call = __builtin_amdgcn_readfirstlane(prev) + 1u;
            const size_t so = OFF_DATA + (call % NSLOT) * SLOT_BYTES;
            char* sb = dst + so + ((size_t)(4 * NX + idx * NY) * 64 + kf) * 1024;
            u32x4 h[NY];
            h[0] = pack8(p[0], q[0]);
            asm volatile("" : "+v"(h[0]));
            wait_until(pos_time<NS_O>(tstart, kf));
            store_wt(sb, lane * 16, h[0]);
            wait_until(tstart + Y_START);
            unsigned tsv;
            issue_ts(tsv, self, lane);
#pragma unroll
            for (int i = 1; i < NY; ++i) issue_x(p[i], q[i], xin + (NX + i) * XB, xo);
#pragma unroll
            for (int i = 1; i < NY; ++i) wait2<0>(p[i], q[i]);
            asm volatile("" : "+v"(tsv));
            const unsigned tref = earliest(tsv, tstart);
#pragma unroll
            for (int i = 1; i < NY; ++i) { h[i] = pack8(p[i], q[i]); asm volatile("" : "+v"(h[i])); }
            STAMP(0);
#pragma unroll
            for (int i = 1; i < NY; ++i) {
                wait_until(pos_time<NS_O>(tref, 256 + i * 64 + kf));
                if (i == NY - 1) STAMP(3);
                store_wt(sb + (size_t)i * 65536, lane * 16, h[i]);
            }
            __hip_atomic_store(cnt, call, __ATOMIC_RELAXED, __HIP_MEMORY_SCOPE_AGENT);
            bool ok = wait_words<VPOLL_SLP, 0u>(self + so + A_BYTES + (size_t)qd * 262144 + (size_t)kf * 4096, 4092, tmo, call);
            ok = ok && wait_words<VPOLL_SLP, HOLE>(outb + (size_t)(tile >> 16) * (NLOC * 4) + qd * 256, 15 * NLOC * 4 + 252, tmo, call);
            STAMP(7);
            const char* tl = outb + (size_t)(tile & 0xffffu) * (NLOC * 4) + qd * 256;
            const unsigned to = (unsigned)((lane >> 4) * (NLOC * 4) + (lane & 15) * 16);
            unsigned spins = 0;
            const unsigned t0 = now32();
            while (ok) {
                u32x4 v0, v1, v2, v3;
                asm volatile("global_load_dwordx4 %0, %4, %6 offset:-4096 sc0 sc1\n\tglobal_load_dwordx4 %1, %4, %6 sc0 sc1\n\t"
                             "global_load_dwordx4 %2, %5, %6 offset:-4096 sc0 sc1\n\tglobal_load_dwordx4 %3, %5, %6 sc0 sc1\n\ts_waitcnt vmcnt(0)"
                             : "=&v"(v0), "=&v"(v1), "=&v"(v2), "=&v"(v3) : "v"(to + 4096), "v"(to + 12288), "s"(tl) : "memory");
                if (__ballot(max(max(max4(v0), max4(v1)), max(max4(v2), max4(v3))) == HOLE) == 0ull) break;
                if (give_up(spins, t0, tmo, call)) ok = false;
            }
        } else {
            if (b < 8) __hip_atomic_store((unsigned*)(self + OFF_TS) + b, tstart, __ATOMIC_RELAXED, __HIP_MEMORY_SCOPE_SYSTEM);
            prev = __hip_atomic_load(cnt, __ATOMIC_RELAXED, __HIP_MEMORY_SCOPE_AGENT);
            const unsigned call = __builtin_amdgcn_readfirstlane(prev) + 1u;
            __hip_atomic_store(cnt, call, __ATOMIC_RELAXED, __HIP_MEMORY_SCOPE_AGENT);
            char* cb = self + OFF_DATA + ((call + NSLOT - 1) % NSLOT) * SLOT_BYTES + (size_t)b * (SLOT_BYTES / 64);
            const u32x4 z = {0u, 0u, 0u, 0u};
#pragma unroll 1
            for (int i = 0; i < (int)(SLOT_BYTES / 64 / 1024); ++i) {
                wait_until(tstart + CLR_START + i * CLR_STEP);
                store_wt(cb + i * 1024, lane * 16, z);
            }
        }
        asm volatile("s_waitcnt vmcnt(0)" ::: "memory");
        STAMP(9);
        return;
    }

    const bool remote = b < 192;
    unsigned call;
    if (remote && wv >= 6) {
        const int u = (b - 64) * 2 + (wv - 6);
        const int kf = u >> 2, qd = u & 3;
        const unsigned trow = a.tiles[kf] & 0xffffu;
        char* wdst = a.link[3 + qd];
        f32x4 w4[8];
        unsigned prev;
        asm volatile("v_mov_b32 %0, 0\n\tglobal_load_dword %0, %0, %1 sc1" : "=&v"(prev) : "s"(cnt) : "memory");
        {
            const float* wb = win + (size_t)kf * 32 * NLOC + qd * 64;
            const unsigned wo = (unsigned)(((lane >> 4) * 8 * NLOC + (lane & 15) * 4) * 4 + 4096);
            asm volatile("global_load_dwordx4 %0, %8, %9 offset:-4096\n\tglobal_load_dwordx4 %1, %8, %9 offset:-3072\n\t"
                         "global_load_dwordx4 %2, %8, %9 offset:-2048\n\tglobal_load_dwordx4 %3, %8, %9 offset:-1024\n\t"
                         "global_load_dwordx4 %4, %8, %9 offset:0\n\tglobal_load_dwordx4 %5, %8, %9 offset:1024\n\t"
                         "global_load_dwordx4 %6, %8, %9 offset:2048\n\tglobal_load_dwordx4 %7, %8, %9 offset:3072"
                         : "=&v"(w4[0]), "=&v"(w4[1]), "=&v"(w4[2]), "=&v"(w4[3]), "=&v"(w4[4]), "=&v"(w4[5]), "=&v"(w4[6]), "=&v"(w4[7])
                         : "v"(wo), "s"(wb) : "memory");
        }
        {
            char* tl = outb + (size_t)trow * (NLOC * 4) + qd * 256;
            const unsigned to = (unsigned)((lane >> 4) * (NLOC * 4) + (lane & 15) * 16);
            const u32x4 hole = {HOLE, HOLE, HOLE, HOLE};
#pragma unroll
            for (int j = 0; j < 4; ++j) store_wt(tl + j * 4096, to, hole);
        }
        asm volatile("s_waitcnt vmcnt(4)" : "+v"(prev), "+v"(w4[0]), "+v"(w4[1]), "+v"(w4[2]), "+v"(w4[3]), "+v"(w4[4]), "+v"(w4[5]), "+v"(w4[6]), "+v"(w4[7]) :: "memory");
        call = __builtin_amdgcn_readfirstlane(prev) + 1u;
        unsigned tsv;
        issue_ts(tsv, self, lane);
        u32x4 hw[4];
#pragma unroll
        for (int f = 0; f < 4; ++f) {
#pragma unroll
            for (int j = 0; j < 4; ++j) hw[f][j] = pack2(w4[2 * j][f] * WSCALE_F, w4[2 * j + 1][f] * WSCALE_F);
            asm volatile("" : "+v"(hw[f]));
        }
        const size_t bo = OFF_DATA + (call % NSLOT) * SLOT_BYTES + A_BYTES + (size_t)kf * 4096;
        asm volatile("s_waitcnt vmcnt(0)" : "+v"(tsv) :: "memory");
        const unsigned tref = earliest(tsv, tstart);
        STAMP(4);
#pragma unroll
        for (int f = 0; f < 4; ++f) store_wt(self + bo + (size_t)qd * 262144 + f * 1024, lane * 16, hw[f]);
#pragma unroll
        for (int f = 0; f < 4; ++f) {
            wait_until(pos_time<NS_O>(tref, 64 + 4 * kf + f));
            if (f == 3) STAMP(5);
            store_wt(wdst + bo + (size_t)(4 + idx) * 262144 + f * 1024, lane * 16, hw[f]);
        }
    } else {
        call = __builtin_amdgcn_readfirstlane(__hip_atomic_load(cnt, __ATOMIC_RELAXED, __HIP_MEMORY_SCOPE_AGENT)) + 1u;
    }
    __hip_atomic_store(cnt, call, __ATOMIC_RELAXED, __HIP_MEMORY_SCOPE_AGENT);
    const int xc = b & 7, j = ((b - 64) >> 3) & 15;
    const int cg = (xc & 1) * 2 + (j & 1) + (remote ? 4 : 0);
    const int pos0 = (xc >> 1) + 4 * (j >> 1);
    const int nit = remote ? 2 : 4;
    const int pstep = remote ? 32 : 16;
    const char* slotp = self + OFF_DATA + (call % NSLOT) * SLOT_BYTES;
    char* ob = (char*)a.outp[cg];
    const unsigned va = lane * 16 + 4096;
    STAMP(0);
    bool ok = true;
    u32x4 bf[32];
    unsigned tsv;
    {
        const char* bb = slotp + A_BYTES + (size_t)cg * 262144 + (size_t)wv * 32768;
        if (remote) ok = wait_words<BPOLL_SLP, 0u>(bb, 32764, tmo, call);
        ok = ok && wait_words<BPOLL_SLP, 0u>(bb, (lane & 7) * 4096 + 4092, tmo, call);
        STAMP(6);
        issue_ts(tsv, self, lane);
#pragma unroll
        for (int g = 0; g < 4; ++g) issue8(&bf[g * 8], bb + g * 8192, va);
        unsigned pend = 0;
        wait8<24>(&bf[0]);
        asm volatile("" : "+v"(tsv));
        if (!arrived8(&bf[0])) pend |= 1u;
        wait8<16>(&bf[8]);
        if (!arrived8(&bf[8])) pend |= 2u;
        wait8<8>(&bf[16]);
        if (!arrived8(&bf[16])) pend |= 4u;
        wait8<0>(&bf[24]);
        if (!arrived8(&bf[24])) pend |= 8u;
        if (__builtin_expect(pend != 0u, 0)) {
            unsigned spins = 0;
            const unsigned t0 = now32();
            while (ok && pend != 0u) {
#pragma unroll
                for (int g = 0; g < 4; ++g)
                    if ((pend >> g) & 1u) {
                        load8(&bf[g * 8], 255u, bb + g * 8192, va);
                        if (arrived8(&bf[g * 8])) pend &= ~(1u << g);
                    }
                if (give_up(spins, t0, tmo, call)) ok = false;
            }
        }
    }
    STAMP(1);
    unsigned tout = pos_time<NS_O>(earliest(tsv, tstart), OUT_POS);
    {
        const unsigned tb = now32() + TB_GAP;
        if ((int)(tb - tout) > 0) tout = tb;
    }
    const int pofs = (wv * 16 + (lane & 15)) * PPITCH + (lane >> 4) * 16;
    const int erow = wv * 2 + (lane >> 5);
    const int eofs = erow * PPITCH + (lane & 31) * 2;
    const unsigned vo = (unsigned)(erow * NLOC * 4 + (lane & 31) * 8);
    u32x2 o = {0u, 0u};
    char* oaddr = ob;
    unsigned otime = 0;
    bool held = false;
#pragma unroll 1
    for (int it = 0; it < nit; ++it) {
        u32x4 af[8];
        const int pos = pos0 + it * pstep;
        const unsigned item = a.items[pos];
        const char* ab = slotp + (size_t)(item & 255u) * 65536 + (size_t)wv * 8192;
        f32x4 acc[4] = {};
        {
            const int back = min(LEAD, (int)((item >> 24) * 64u) + wv * 8 + 7);
            const char* sp = ab + (7 - back) * 1024;
            unsigned spins = 0;
            const unsigned t0 = now32();
            while (ok) {
                if (held && (int)(now32() - otime) >= 0) { store_wt2(oaddr, vo, o); held = false; STAMP(11); }
                unsigned v;
                asm volatile("global_load_dword %0, %1, %2 sc0 sc1\n\ts_waitcnt vmcnt(0)" : "=&v"(v) : "v"(1020), "s"(sp) : "memory");
                if (__builtin_amdgcn_readfirstlane(v) != 0u) break;
                if (give_up(spins, t0, tmo, call)) ok = false;
                __builtin_amdgcn_s_sleep(POLL_SLP);
            }
        }
        STAMP(8 + it);
        unsigned pend = 255u;
#define STEP(i) if ((pend >> i) & 1u) { if (arrived(af[i])) { pend &= ~(1u << i); \
            _Pragma("unroll") for (int ff = 0; ff < 4; ++ff) acc[ff] = __builtin_amdgcn_mfma_f32_16x16x32_f16( \
                __builtin_bit_cast(f16x8, bf[i * 4 + ff]), __builtin_bit_cast(f16x8, af[i]), acc[ff], 0, 0, 0); } }
        issue8(af, ab, va);
        wait_frag<7>(af[0]); STEP(0) wait_frag<6>(af[1]); STEP(1) wait_frag<5>(af[2]); STEP(2) wait_frag<4>(af[3]); STEP(3)
        wait_frag<3>(af[4]); STEP(4) wait_frag<2>(af[5]); STEP(5) wait_frag<1>(af[6]); STEP(6) wait_frag<0>(af[7]); STEP(7)
        if (pend != 0u) {
            unsigned spins = 0;
            const unsigned t0 = now32();
            while (ok && pend != 0u) {
                load8(af, pend, ab, va);
                STEP(0) STEP(1) STEP(2) STEP(3) STEP(4) STEP(5) STEP(6) STEP(7)
                if (give_up(spins, t0, tmo, call)) ok = false;
            }
        }
        STAMP(2 + it);
        if (!ok) { float nn; asm volatile("v_mov_b32 %0, 0x7fc00000" : "=v"(nn)); acc[0][0] = nn; }
        float* pb = &part[(it & 1) * 8 * 16 * PPITCH];
#pragma unroll
        for (int i = 0; i < 4; ++i) {
            f32x4 q0 = {acc[0][i], acc[1][i], acc[2][i], acc[3][i]};
            *(f32x4*)(pb + pofs + i * 4) = q0;
        }
        asm volatile("s_waitcnt lgkmcnt(0)\n\ts_barrier" ::: "memory");
        f32x2 t[8];
#pragma unroll
        for (int w = 0; w < 8; ++w) t[w] = *(const f32x2*)(pb + eofs + w * 16 * PPITCH);
        asm volatile("" : "+v"(t[0]), "+v"(t[1]), "+v"(t[2]), "+v"(t[3]), "+v"(t[4]), "+v"(t[5]), "+v"(t[6]), "+v"(t[7]));
        const f32x2 s = ((t[0] + t[1]) + (t[2] + t[3])) + ((t[4] + t[5]) + (t[6] + t[7]));
        u32x2 on = {gelu_bits(s[0] * (1.0f / WSCALE_F)), gelu_bits(s[1] * (1.0f / WSCALE_F))};
        asm volatile("" : "+v"(on));
        if (held) { wait_until(otime); store_wt2(oaddr, vo, o); STAMP(11); }
        o = on;
        oaddr = ob + (size_t)((item >> 8) & 0xffffu) * (NLOC * 4);
        otime = remote ? tout + (unsigned)(((it == 0 ? (pos + ROT) & 31 : pos) * 8 + wv) * NS_OUT) / 200u : tstart;
        held = true;
    }
    wait_until(otime);
    store_wt2(oaddr, vo, o);
    STAMP(7);
    asm volatile("s_waitcnt vmcnt(0)" ::: "memory");
}

extern "C" void kernel_launch(int rank, int nranks, ncclComm_t comm, hipStream_t stream,
                              void* const* d_in, const int* in_sizes, int n_in,
                              void* d_out, int out_size, const int* in_dtypes, int out_dtype,
                              void* const* d_out_all, void* const* d_ws_all, size_t ws_bytes) {
    if (nranks != NR || (unsigned)rank >= (unsigned)NR || ws_bytes < WS_NEED) { fprintf(stderr, "bad config\n"); return; }
    Args a;
    const int T = rank >> 2, idx = rank & 3;
    a.x = (const float*)d_in[0];
    a.w = (const float*)d_in[1];
    a.out = (float*)d_out;
    for (int d = 1; d < 4; ++d) a.link[d - 1] = (char*)d_ws_all[T * 4 + ((idx + d) & 3)];
    for (int q = 0; q < 4; ++q) a.link[3 + q] = (char*)d_ws_all[(T ^ 1) * 4 + q];
    a.link[7] = a.self = (char*)d_ws_all[rank];
    for (int q = 0; q < 4; ++q) {
        a.outp[q] = (float*)d_out + 64 * q;
        a.outp[4 + q] = (float*)d_out_all[(T ^ 1) * 4 + q] + 64 * idx;
    }
    int key[64]; unsigned val[64]; int n = 0;
    for (int s = 0; s < 4; ++s)
        for (int i = 0; i < NX; ++i) {
            key[n] = s == idx ? 0 : i + 1;
            val[n] = (unsigned)(s * NX + i) | (unsigned)((T * 4 + s) * 256 + i * 16) << 8 | (unsigned)i << 24;
            const unsigned r = (unsigned)(((T ^ 1) * 4 + s) * 256 + i * 16);
            a.tiles[n++] = r | (i > 0 ? r - 16 : r) << 16;
        }
    for (int s = 0; s < 4; ++s)
        for (int i = 0; i < NY; ++i) {
            key[n] = i == 0 ? 1 : 5 + i;
            val[n] = (unsigned)(4 * NX + s * NY + i) | (unsigned)(((T ^ 1) * 4 + s) * 256 + (NX + i) * 16) << 8 | (unsigned)(i > 1 ? i - 1 : 0) << 24;
            const unsigned r = (unsigned)((T * 4 + s) * 256 + (NX + i) * 16);
            a.tiles[n++] = r | (i > 0 ? r - 16 : r) << 16;
        }
    int pos = 0;
    for (int k = 0; k <= 32; ++k)
        for (int j = 0; j < 64; ++j)
            if (key[j] == k) a.items[pos++] = val[j];
    a.idx = idx;
    a.pad = 0;
    ag_gemm<<<NWG, NTHR, 0, stream>>>(a);
    HIP_CHECK(hipGetLastError());
}
```

```cpp
#include <hip/hip_runtime.h>
#include <rccl/rccl.h>
#include <cstdio>
#include <algorithm>
#include <vector>

#define HIP_CHECK(expr) do { hipError_t _e = (expr); if (_e != hipSuccess) { \
    fprintf(stderr, "HIP error %s at %s:%d\n", hipGetErrorString(_e), __FILE__, __LINE__); \
    return; } } while (0)

typedef unsigned u32x4 __attribute__((ext_vector_type(4)));
typedef unsigned u32x2 __attribute__((ext_vector_type(2)));
typedef float f32x4 __attribute__((ext_vector_type(4)));
typedef float f32x2 __attribute__((ext_vector_type(2)));
typedef _Float16 f16x8 __attribute__((ext_vector_type(8)));
typedef _Float16 f16x2 __attribute__((ext_vector_type(2)));

#define NX 11
#define NS_X 177
#define NS_O 177
#define NS_OUT 175
#define NS_LATE 215
#define VDEPTH 3
#define VGAP 4
#define T_BASE 50
#define LEAD 8
#define POLL_SLP 1
#define BPOLL_SLP 2
#define VPOLL_SLP 2
#define CLR_START 400
#define CLR_STEP 9
#define TS_READ 50
#define WPOS 40
#define OUT_POS (256 + NY * 64)
#define TB_GAP 60
#define ROT 6
#define XMAP 1
constexpr int NR = 8;
constexpr int NY = 16 - NX;
constexpr int KDIM = 2048, NLOC = 256;
constexpr int NWG = 256, NTHR = 512;
constexpr size_t OFF_CNT = 0;
constexpr size_t OFF_TMO = 32768;
constexpr size_t OFF_TS = 33024;
constexpr size_t OFF_DIAG = 65536;
constexpr size_t OFF_DATA = 1u << 20;
constexpr size_t A_BYTES = 64u * 65536u;
constexpr size_t B_BYTES = 8u * 262144u;
constexpr size_t SLOT_BYTES = A_BYTES + B_BYTES;
constexpr int NSLOT = 3;
constexpr size_t WS_NEED = OFF_DATA + NSLOT * SLOT_BYTES;
constexpr unsigned DEADLINE_TICKS = 50000000u;
constexpr float WSCALE_F = 64.0f;
constexpr int PPITCH = 68;
constexpr unsigned HOLE = 0xffffffffu;

struct Wave {
    const char* src;
    unsigned kstep;
    unsigned step;
    unsigned off0, off2;
    char* link;
};
struct Args {
    Wave wave[8];
    char* self;
    float* out;
    int idx;
    int pad;
    float* outp[8];
    unsigned items[64];
    unsigned tiles[64];
};

__device__ __forceinline__ void store_wt(char* sbase, unsigned voff, u32x4 v) {
    asm volatile("global_store_dwordx4 %0, %1, %2 sc0 sc1\n\ts_nop 1" :: "v"(voff), "v"(v), "s"(sbase) : "memory");
}
__device__ __forceinline__ void store_wt2(char* sbase, unsigned voff, u32x2 v) {
    asm volatile("global_store_dwordx2 %0, %1, %2 sc0 sc1\n\ts_nop 1" :: "v"(voff), "v"(v), "s"(sbase) : "memory");
}
__device__ __forceinline__ unsigned pack2(float a, float b) {
    f16x2 h = {(_Float16)a, (_Float16)b};
    unsigned u = __builtin_bit_cast(unsigned, h);
    return u == 0u ? 0x80000000u : u;
}
__device__ __forceinline__ u32x4 pack8(f32x4 a, f32x4 b) {
    u32x4 r = {pack2(a[0], a[1]), pack2(a[2], a[3]), pack2(b[0], b[1]), pack2(b[2], b[3])};
    return r;
}
__device__ __forceinline__ unsigned gelu_bits(float y) {
    const float k0 = -2.0f * 0.7978845608028654f * 1.4426950408889634f;
    const float k1 = k0 * 0.044715f;
    float e = __builtin_amdgcn_exp2f(y * (k0 + k1 * y * y));
    unsigned u = __builtin_bit_cast(unsigned, y * __builtin_amdgcn_rcpf(1.0f + e));
    return u == HOLE ? 0x7fc00000u : u;
}
__device__ __forceinline__ unsigned long long now() { return __builtin_amdgcn_s_memrealtime(); }
__device__ __forceinline__ unsigned now32() { return (unsigned)__builtin_amdgcn_s_memrealtime(); }
__device__ __forceinline__ bool give_up(unsigned& spins, unsigned t0, unsigned* tmo, unsigned call) {
    if (__builtin_expect((++spins & 63u) != 0u, 1)) return false;
    unsigned t, u;
    asm volatile("v_mov_b32 %0, 0\n\tglobal_load_dword %0, %0, %1 sc1\n\ts_waitcnt vmcnt(0)" : "=&v"(t) : "s"(tmo) : "memory");
    if (__builtin_amdgcn_readfirstlane(t) == call || now32() - t0 > DEADLINE_TICKS) {
        asm volatile("v_mov_b32 %0, 0\n\tv_mov_b32 %1, %3\n\tglobal_store_dword %0, %1, %2 sc1\n\ts_nop 1"
                     : "=&v"(t), "=&v"(u) : "s"(tmo), "s"(call) : "memory");
        return true;
    }
    return false;
}
__device__ __forceinline__ void wait_until(unsigned t) {
    while ((int)(now32() - t) < 0) __builtin_amdgcn_s_sleep(1);
}
template <int NS> __device__ __forceinline__ unsigned pos_time(unsigned tstart, int pos) { return tstart + T_BASE + (unsigned)(pos * NS) / 100u; }

__device__ __forceinline__ void issue_ts(unsigned& v, const char* self, int lane) {
    asm volatile("global_load_dword %0, %1, %2 sc0 sc1" : "=&v"(v) : "v"((unsigned)(lane & 7) * 4u), "s"(self + OFF_TS) : "memory");
}
__device__ __forceinline__ unsigned earliest(unsigned v, unsigned tstart) {
    unsigned d = tstart - v;
    d = d < 100u ? d : 0u;
    unsigned m = 0;
#pragma unroll
    for (int i = 0; i < 8; ++i) m = max(m, (unsigned)__builtin_amdgcn_readlane(d, i));
    return tstart - m;
}
constexpr unsigned L_CNT = 16384, L_TREF = 16388, L_TOUT = 2 * 8 * 16 * PPITCH * 4;
__device__ __forceinline__ void put_frag(int blk, unsigned lw, f32x4 p, f32x4 q) {
    u32x2 a = {pack2(p[0], p[1]), pack2(p[2], p[3])}, c = {pack2(q[0], q[1]), pack2(q[2], q[3])};
    asm volatile("ds_write_b64 %0, %1\n\tds_write_b64 %0, %2 offset:128" :: "v"(lw + blk * 1024), "v"(a), "v"(c) : "memory");
}
__device__ __forceinline__ void get_frag(u32x4& h, int blk, unsigned lr) {
    asm volatile("ds_read_b128 %0, %1 offset:%2" : "=&v"(h) : "v"(lr), "n"(blk * 1024) : "memory");
}
__device__ __forceinline__ void lds_w32(unsigned ad, unsigned v) { asm volatile("ds_write_b32 %0, %1" :: "v"(ad), "v"(v) : "memory"); }
__device__ __forceinline__ unsigned lds_r32(unsigned ad) {
    unsigned v;
    asm volatile("ds_read_b32 %0, %1\n\ts_waitcnt lgkmcnt(0)" : "=&v"(v) : "v"(ad) : "memory");
    return __builtin_amdgcn_readfirstlane(v);
}
__device__ __forceinline__ void lds_add(unsigned ad, unsigned v) {
    unsigned long long e;
    asm volatile("s_waitcnt lgkmcnt(0)\n\ts_mov_b64 %0, exec\n\ts_mov_b64 exec, 1\n\tds_add_u32 %1, %2\n\ts_mov_b64 exec, %0"
                 : "=&s"(e) : "v"(ad), "v"(v) : "memory");
}
__device__ __forceinline__ void lds_wait(unsigned want, unsigned mask) {
    while ((lds_r32(L_CNT) & mask) != want) __builtin_amdgcn_s_sleep(1);
}
template <int N> __device__ __forceinline__ void wait2(f32x4& p, f32x4& q) {
    asm volatile("s_waitcnt vmcnt(%2)" : "+v"(p), "+v"(q) : "n"(N) : "memory");
}

#define LD1(i, off) "s_bitcmp1_b32 %[pend], " #i "\n\ts_cbranch_scc0 Lx" #i "_%=\n\tglobal_load_dwordx4 %" #i ", %[va], %[sb] offset:" #off " sc0 sc1\nLx" #i "_%=:\n\t"
__device__ __forceinline__ void load8(u32x4* r, unsigned pend, const char* sb, unsigned va) {
    asm volatile("s_nop 4\n\t"
                 LD1(0, -4096) LD1(1, -3072) LD1(2, -2048) LD1(3, -1024) LD1(4, 0) LD1(5, 1024) LD1(6, 2048) LD1(7, 3072)
                 "s_waitcnt vmcnt(0)"
                 : "+v"(r[0]), "+v"(r[1]), "+v"(r[2]), "+v"(r[3]), "+v"(r[4]), "+v"(r[5]), "+v"(r[6]), "+v"(r[7])
                 : [pend] "s"(pend), [va] "v"(va), [sb] "s"(sb)
                 : "memory", "scc");
}
__device__ __forceinline__ void issue8(u32x4* r, const char* sb, unsigned va) {
    asm volatile("s_nop 4\n\t"
                 "global_load_dwordx4 %0, %[va], %[sb] offset:-4096 sc0 sc1\n\t"
                 "global_load_dwordx4 %1, %[va], %[sb] offset:-3072 sc0 sc1\n\t"
                 "global_load_dwordx4 %2, %[va], %[sb] offset:-2048 sc0 sc1\n\t"
                 "global_load_dwordx4 %3, %[va], %[sb] offset:-1024 sc0 sc1\n\t"
                 "global_load_dwordx4 %4, %[va], %[sb] offset:0 sc0 sc1\n\t"
                 "global_load_dwordx4 %5, %[va], %[sb] offset:1024 sc0 sc1\n\t"
                 "global_load_dwordx4 %6, %[va], %[sb] offset:2048 sc0 sc1\n\t"
                 "global_load_dwordx4 %7, %[va], %[sb] offset:3072 sc0 sc1"
                 : "=&v"(r[0]), "=&v"(r[1]), "=&v"(r[2]), "=&v"(r[3]), "=&v"(r[4]), "=&v"(r[5]), "=&v"(r[6]), "=&v"(r[7])
                 : [va] "v"(va), [sb] "s"(sb)
                 : "memory");
}
template <int N> __device__ __forceinline__ void wait_frag(u32x4& r) {
    asm volatile("s_waitcnt vmcnt(%1)" : "+v"(r) : "n"(N) : "memory");
}
template <int N> __device__ __forceinline__ void wait8(u32x4* r) {
    asm volatile("s_waitcnt vmcnt(%8)"
                 : "+v"(r[0]), "+v"(r[1]), "+v"(r[2]), "+v"(r[3]), "+v"(r[4]), "+v"(r[5]), "+v"(r[6]), "+v"(r[7]) : "n"(N) : "memory");
}
__device__ __forceinline__ unsigned min4(u32x4 v) { return min(min(v[0], v[1]), min(v[2], v[3])); }
__device__ __forceinline__ unsigned max4(u32x4 v) { return max(max(v[0], v[1]), max(v[2], v[3])); }
__device__ __forceinline__ bool arrived(u32x4 v) { return __ballot(min4(v) == 0u) == 0ull; }
__device__ __forceinline__ bool arrived8(const u32x4* r) {
    unsigned m = min4(r[0]);
#pragma unroll
    for (int i = 1; i < 8; ++i) m = min(m, min4(r[i]));
    return __ballot(m == 0u) == 0ull;
}

__device__ __forceinline__ void issue_tile(u32x4* v, const char* tl, unsigned to) {
    asm volatile("global_load_dwordx4 %0, %4, %6 offset:-4096 sc0 sc1\n\tglobal_load_dwordx4 %1, %4, %6 sc0 sc1\n\t"
                 "global_load_dwordx4 %2, %5, %6 offset:-4096 sc0 sc1\n\tglobal_load_dwordx4 %3, %5, %6 sc0 sc1"
                 : "=&v"(v[0]), "=&v"(v[1]), "=&v"(v[2]), "=&v"(v[3]) : "v"(to + 4096), "v"(to + 12288), "s"(tl) : "memory");
}
template <int SLP, unsigned EMPTY>
__device__ __forceinline__ bool wait_words(const char* sb, unsigned vo, unsigned* tmo, unsigned call) {
    unsigned spins = 0;
    const unsigned t0 = now32();
    while (true) {
        unsigned v;
        asm volatile("global_load_dword %0, %1, %2 sc0 sc1\n\ts_waitcnt vmcnt(0)" : "=&v"(v) : "v"(vo), "s"(sb) : "memory");
        if (__ballot(v == EMPTY) == 0ull) return true;
        if (give_up(spins, t0, tmo, call)) return false;
        __builtin_amdgcn_s_sleep(SLP);
    }
}

#define FINISH() do { asm volatile("s_waitcnt vmcnt(0)" ::: "memory"); __builtin_amdgcn_endpgm(); } while (0)
#define STAMP(i) do {} while (0)

__global__ __launch_bounds__(NTHR, 2) void ag_gemm(Args a) {
    __shared__ __attribute__((aligned(16))) float part[2 * 8 * 16 * PPITCH + 4];
    const int tid = threadIdx.x;
    const int lane = tid & 63;
    const int wv = __builtin_amdgcn_readfirstlane(tid >> 6);
    const int b = blockIdx.x;
    const Wave wt = a.wave[wv];
    char* self = a.self;
    char* dst = wt.link;
    char* outb = (char*)a.out;
    const int idx = a.idx;
    const unsigned tile = a.tiles[b & 63];
    asm volatile("" :: "s"(self), "s"(dst), "s"(wt.src), "s"(wt.kstep), "s"(wt.step), "s"(wt.off0), "s"(wt.off2), "s"(idx), "s"(outb), "s"(tile));
    unsigned* cnt = (unsigned*)(self + OFF_CNT + b * 128 + wv * 4);
    unsigned* tmo = (unsigned*)(self + OFF_TMO);

    if (__builtin_expect(b < 64, 1)) {
        const int kf = b;
        if (b < 8 && wv == 7) __hip_atomic_store((unsigned*)(self + OFF_TS) + b, now32(), __ATOMIC_RELAXED, __HIP_MEMORY_SCOPE_SYSTEM);
        unsigned prev;
        unsigned long long t64;
        f32x4 r[8];
        {
            const char* lb = wt.src;
            const unsigned st = wt.step;
            const unsigned sh = wt.kstep & 31u;
            const unsigned ol = (((unsigned)lane >> sh) << 13) + (((unsigned)lane & ((1u << sh) - 1u)) << 4) + (unsigned)kf * (wt.kstep & ~31u);
            const unsigned o0 = ol + wt.off0, o2 = ol + wt.off2;
            asm volatile("s_memrealtime %9\n\tv_mov_b32 %8, 0\n\tglobal_load_dword %8, %8, %[cnt] sc1\n\ts_cmp_lt_u32 %[wv], 3\n\ts_cbranch_scc0 La_%=\n\t"
                         "global_load_dwordx4 %0, %[a0], %[lb]\n\tglobal_load_dwordx4 %1, %[a1], %[lb]\n"
                         "La_%=:\n\ts_barrier\n\ts_cbranch_scc1 Lb_%=\n\t"
                         "global_load_dwordx4 %0, %[a0], %[lb]\n\tglobal_load_dwordx4 %1, %[a1], %[lb]\n"
                         "Lb_%=:\n\t"
                         "global_load_dwordx4 %2, %[a2], %[lb]\n\tglobal_load_dwordx4 %3, %[a3], %[lb]\n\t"
                         "global_load_dwordx4 %4, %[a4], %[lb]\n\tglobal_load_dwordx4 %5, %[a5], %[lb]\n\t"
                         "global_load_dwordx4 %6, %[a6], %[lb]\n\tglobal_load_dwordx4 %7, %[a7], %[lb]\n\ts_waitcnt lgkmcnt(0)"
                         : "=&v"(r[0]), "=&v"(r[1]), "=&v"(r[2]), "=&v"(r[3]), "=&v"(r[4]), "=&v"(r[5]), "=&v"(r[6]), "=&v"(r[7]), "=&v"(prev), "=&s"(t64)
                         : [wv] "s"(wv), [lb] "s"(lb), [cnt] "s"(cnt), [a0] "v"(o0), [a1] "v"(o0 + st), [a2] "v"(o2), [a3] "v"(o2 + st),
                           [a4] "v"(o2 + 2 * st), [a5] "v"(o2 + 3 * st), [a6] "v"(o2 + 4 * st), [a7] "v"(o2 + 5 * st)
                         : "memory", "scc");
        }
        const unsigned tstart = (unsigned)t64;
        const int first = wv == 0 ? 0 : wv == 1 ? NX : wv == 2 ? 1 : NX > 11 ? 11 : 12;
        const int rest = wv < 3 ? 2 + 3 * wv : 13;
        const unsigned lw = (unsigned)(((((lane & 7) >> 1) * 16 + (lane >> 3)) * 16 + (lane & 1) * 8));
        const unsigned lr = (unsigned)(lane * 16);
        if (wv < 3) {
            wait2<6>(r[0], r[1]);
            put_frag(first, lw, r[0], r[1]);
            asm volatile("s_waitcnt lgkmcnt(0)\n\ts_barrier" ::: "memory");
            u32x4 h[NX];
            get_frag(h[0], 0, lr);
            asm volatile("s_waitcnt vmcnt(6) lgkmcnt(0)" : "+v"(prev), "+v"(h[0]) :: "memory");
            const unsigned call = __builtin_amdgcn_readfirstlane(prev) + 1u;
            char* sb = dst + OFF_DATA + (call % NSLOT) * SLOT_BYTES + ((size_t)(idx * NX) * 64 + kf) * 1024;
            STAMP(0);
            wait_until(pos_time<NS_X>(tstart, kf));
            STAMP(1);
            store_wt(sb, lr, h[0]);
#pragma unroll
            for (int i = 1; i < 4; ++i) wait2<1>(r[2 * i], r[2 * i + 1]);
#pragma unroll
            for (int i = 1; i < 4; ++i) put_frag(rest + i - 1, lw, r[2 * i], r[2 * i + 1]);
            lds_add(L_CNT, 1u);
            lds_wait(0x104u, 0x1ffu);
            const unsigned tref = lds_r32(L_TREF);
#pragma unroll
            for (int i = 1; i < NX; ++i) get_frag(h[i], i, lr);
#pragma unroll
            for (int i = 1; i < NX; ++i) asm volatile("s_waitcnt lgkmcnt(0)" : "+v"(h[i]) :: "memory");
            STAMP(2);
#pragma unroll
            for (int i = 1; i < NX; ++i) {
                wait_until(pos_time<NS_X>(tref, i * 64 + kf));
                if (i == NX - 1) STAMP(6);
                store_wt(sb + (size_t)i * 65536, lr, h[i]);
            }
            __hip_atomic_store(cnt, call, __ATOMIC_RELAXED, __HIP_MEMORY_SCOPE_AGENT);
            FINISH();
        } else if (wv < 7) {
            const int qd = wv - 3;
            char* tl = outb + (size_t)(tile & 0xffffu) * (NLOC * 4) + qd * 256;
            const unsigned to = (unsigned)((lane >> 4) * (NLOC * 4) + (lane & 15) * 16);
            {
                const u32x4 hole = {HOLE, HOLE, HOLE, HOLE};
#pragma unroll
                for (int j = 0; j < 4; ++j) store_wt(tl + j * 4096, to, hole);
            }
            asm volatile("s_barrier" ::: "memory");
            u32x4 h[NY];
            get_frag(h[0], NX, lr);
            asm volatile("s_waitcnt vmcnt(12) lgkmcnt(0)" : "+v"(prev), "+v"(h[0]) :: "memory");
            const unsigned call = __builtin_amdgcn_readfirstlane(prev) + 1u;
            const size_t so = OFF_DATA + (call % NSLOT) * SLOT_BYTES;
            char* sb = dst + so + ((size_t)(4 * NX + idx * NY) * 64 + kf) * 1024;
            if (kf < WPOS) wait_until(pos_time<NS_O>(tstart, kf));
            asm volatile("s_cmp_lt_u32 %[kf], %[wp]\n\ts_cbranch_scc0 Lc_%=\n\tglobal_store_dwordx4 %[lr], %[h], %[sb] sc0 sc1\n\t"
                         "s_waitcnt vmcnt(1)\n\ts_branch Ld_%=\nLc_%=:\n\ts_waitcnt vmcnt(0)\nLd_%=:"
                         : "+v"(r[0]), "+v"(r[1]), "+v"(r[2]), "+v"(r[3]), "+v"(r[4]), "+v"(r[5]), "+v"(r[6]), "+v"(r[7])
                         : [kf] "s"(kf), [wp] "n"(WPOS), [lr] "v"(lr), [h] "v"(h[0]), [sb] "s"(sb) : "memory", "scc");
            u32x4 hw[4];
#pragma unroll
            for (int f = 0; f < 4; ++f) {
#pragma unroll
                for (int j = 0; j < 4; ++j) hw[f][j] = pack2(r[2 * j][f] * WSCALE_F, r[2 * j + 1][f] * WSCALE_F);
                asm volatile("" : "+v"(hw[f]));
            }
            const size_t bo = so + A_BYTES + (size_t)kf * 4096;
            STAMP(4);
#pragma unroll
            for (int f = 0; f < 4; ++f) store_wt(self + bo + (size_t)qd * 262144 + f * 1024, lr, hw[f]);
            lds_wait(0x100u, 0x100u);
            const unsigned tref = lds_r32(L_TREF);
#pragma unroll
            for (int f = 0; f < 4; ++f) {
                wait_until(pos_time<NS_O>(tref, WPOS + 4 * kf + f));
                if (f == 3) STAMP(5);
                store_wt(dst + bo + (size_t)(4 + idx) * 262144 + f * 1024, lr, hw[f]);
            }
            lds_wait(0x104u, 0x1ffu);
#pragma unroll
            for (int i = 1; i < NY; ++i) get_frag(h[i], NX + i, lr);
#pragma unroll
            for (int i = 1; i < NY; ++i) asm volatile("s_waitcnt lgkmcnt(0)" : "+v"(h[i]) :: "memory");
            if (kf >= WPOS) {
                wait_until(pos_time<NS_O>(tref, 256 + kf));
                store_wt(sb, lr, h[0]);
            }
#pragma unroll
            for (int i = 1; i < NY; ++i) {
                wait_until(pos_time<NS_O>(tref, 256 + i * 64 + kf));
                if (i == NY - 1) STAMP(3);
                store_wt(sb + (size_t)i * 65536, lr, h[i]);
            }
            __hip_atomic_store(cnt, call, __ATOMIC_RELAXED, __HIP_MEMORY_SCOPE_AGENT);
            bool ok = wait_words<VPOLL_SLP, HOLE>(outb + (size_t)(tile >> 16) * (NLOC * 4) + qd * 256, 15 * NLOC * 4 + 252, tmo, call);
            STAMP(7);
            unsigned spins = 0;
            const unsigned t0 = now32();
            u32x4 v[VDEPTH][4];
#pragma unroll
            for (int d = 0; d < VDEPTH - 1; ++d) { issue_tile(v[d], tl, to); __builtin_amdgcn_s_sleep(VGAP); }
            while (ok) {
                bool done = false;
#pragma unroll
                for (int d = 0; d < VDEPTH; ++d) {
                    issue_tile(v[(d + VDEPTH - 1) % VDEPTH], tl, to);
                    asm volatile("s_waitcnt vmcnt(%4)" : "+v"(v[d][0]), "+v"(v[d][1]), "+v"(v[d][2]), "+v"(v[d][3]) : "n"(4 * (VDEPTH - 1)) : "memory");
                    if (__ballot(max(max(max4(v[d][0]), max4(v[d][1])), max(max4(v[d][2]), max4(v[d][3]))) == HOLE) == 0ull) { done = true; break; }
                    if (VDEPTH > 1) __builtin_amdgcn_s_sleep(VGAP);
                }
                if (done) break;
                if (give_up(spins, t0, tmo, call)) ok = false;
            }
            __builtin_amdgcn_endpgm();
        } else {
            lds_w32(L_CNT, 0u);
            asm volatile("s_waitcnt lgkmcnt(0)\n\ts_barrier" ::: "memory");
            wait_until(tstart + TS_READ);
            unsigned tsv;
            issue_ts(tsv, self, lane);
            asm volatile("s_waitcnt vmcnt(0)" : "+v"(prev), "+v"(tsv) :: "memory");
            lds_w32(L_TREF, earliest(tsv, tstart));
            lds_add(L_CNT, 0x100u);
#pragma unroll
            for (int i = 0; i < 4; ++i) { wait2<0>(r[2 * i], r[2 * i + 1]); put_frag(i == 0 ? first : rest + i - 1, lw, r[2 * i], r[2 * i + 1]); }
            lds_add(L_CNT, 1u);
            const unsigned call = __builtin_amdgcn_readfirstlane(prev) + 1u;
            __hip_atomic_store(cnt, call, __ATOMIC_RELAXED, __HIP_MEMORY_SCOPE_AGENT);
            lds_wait(0x104u, 0x1ffu);
            {
                char* lb = self + OFF_DATA + (call % NSLOT) * SLOT_BYTES + ((size_t)(idx * NX) * 64 + kf) * 1024;
                u32x4 h[NX];
#pragma unroll
                for (int i = 0; i < NX; ++i) get_frag(h[i], i, lr);
#pragma unroll
                for (int i = 0; i < NX; ++i) asm volatile("s_waitcnt lgkmcnt(0)" : "+v"(h[i]) :: "memory");
                STAMP(0);
#pragma unroll
                for (int i = 0; i < NX; ++i) store_wt(lb + (size_t)i * 65536, lr, h[i]);
            }
            char* cb = self + OFF_DATA + ((call + NSLOT - 1) % NSLOT) * SLOT_BYTES + (size_t)b * (SLOT_BYTES / 64);
            const u32x4 z = {0u, 0u, 0u, 0u};
#pragma unroll 1
            for (int i = 0; i < (int)(SLOT_BYTES / 64 / 1024); ++i) {
                wait_until(tstart + CLR_START + i * CLR_STEP);
                store_wt(cb + i * 1024, lr, z);
            }
            FINISH();
        }
    }

    const unsigned tstart = now32();
    const bool remote = b < 192;
    const unsigned call = __builtin_amdgcn_readfirstlane(__hip_atomic_load(cnt, __ATOMIC_RELAXED, __HIP_MEMORY_SCOPE_AGENT)) + 1u;
    __hip_atomic_store(cnt, call, __ATOMIC_RELAXED, __HIP_MEMORY_SCOPE_AGENT);
    const int xc = b & 7, j = ((b - 64) >> 3) & 15;
    const int cg = (xc & 1) * 2 + (j & 1) + (remote ? 4 : 0);
    const int pos0 = (xc >> 1) + 4 * (j >> 1);
    const int nit = remote ? 2 : 4;
    const int pstep = remote ? 32 : 16;
    const char* slotp = self + OFF_DATA + (call % NSLOT) * SLOT_BYTES;
    char* ob = (char*)a.outp[cg];
    const unsigned va = lane * 16 + 4096;
    STAMP(0);
    bool ok = true;
    u32x4 bf[32];
    if (remote && wv == 0) {
        unsigned tsv;
        wait_until(tstart + TS_READ);
        issue_ts(tsv, self, lane);
        asm volatile("s_waitcnt vmcnt(0)" : "+v"(tsv) :: "memory");
        lds_w32(L_TOUT, pos_time<NS_O>(earliest(tsv, tstart), OUT_POS));
    }
    {
        const char* bb = slotp + A_BYTES + (size_t)cg * 262144 + (size_t)wv * 32768;
        if (remote) ok = wait_words<BPOLL_SLP, 0u>(bb, 32764, tmo, call);
        ok = ok && wait_words<BPOLL_SLP, 0u>(bb, (lane & 7) * 4096 + 4092, tmo, call);
        STAMP(6);
#pragma unroll
        for (int g = 0; g < 4; ++g) issue8(&bf[g * 8], bb + g * 8192, va);
        unsigned pend = 0;
        wait8<24>(&bf[0]);
        if (!arrived8(&bf[0])) pend |= 1u;
        wait8<16>(&bf[8]);
        if (!arrived8(&bf[8])) pend |= 2u;
        wait8<8>(&bf[16]);
        if (!arrived8(&bf[16])) pend |= 4u;
        wait8<0>(&bf[24]);
        if (!arrived8(&bf[24])) pend |= 8u;
        if (__builtin_expect(pend != 0u, 0)) {
            unsigned spins = 0;
            const unsigned t0 = now32();
            while (ok && pend != 0u) {
#pragma unroll
                for (int g = 0; g < 4; ++g)
                    if ((pend >> g) & 1u) {
                        load8(&bf[g * 8], 255u, bb + g * 8192, va);
                        if (arrived8(&bf[g * 8])) pend &= ~(1u << g);
                    }
                if (give_up(spins, t0, tmo, call)) ok = false;
            }
        }
    }
    STAMP(1);
    const unsigned tlate = now32() + TB_GAP;
    unsigned tout = 0;
    const int pofs = (wv * 16 + (lane & 15)) * PPITCH + (lane >> 4) * 16;
    const int erow = wv * 2 + (lane >> 5);
    const int eofs = erow * PPITCH + (lane & 31) * 2;
    const unsigned vo = (unsigned)(erow * NLOC * 4 + (lane & 31) * 8);
    u32x2 o = {0u, 0u};
    char* oaddr = ob;
    unsigned otime = 0;
    bool held = false;
#pragma unroll 1
    for (int it = 0; it < nit; ++it) {
        u32x4 af[8];
        const int pos = pos0 + it * pstep;
        const unsigned item = a.items[pos];
        const char* ab = slotp + (size_t)(item & 255u) * 65536 + (size_t)wv * 8192;
        f32x4 acc[4] = {};
        {
            const int back = min(LEAD, (item >> 31) ? 7 : (int)((item >> 24) * 64u) + wv * 8 + 7);
            const char* sp = ab + (7 - back) * 1024;
            unsigned spins = 0;
            const unsigned t0 = now32();
            while (ok) {
                if (held && (int)(now32() - otime) >= 0) { store_wt2(oaddr, vo, o); held = false; STAMP(11); }
                unsigned v;
                asm volatile("global_load_dword %0, %1, %2 sc0 sc1\n\ts_waitcnt vmcnt(0)" : "=&v"(v) : "v"(1020), "s"(sp) : "memory");
                if (__builtin_amdgcn_readfirstlane(v) != 0u) break;
                if (give_up(spins, t0, tmo, call)) ok = false;
                __builtin_amdgcn_s_sleep(POLL_SLP);
            }
        }
        STAMP(8 + it);
        unsigned pend = 255u;
#define STEP(i) if ((pend >> i) & 1u) { if (arrived(af[i])) { pend &= ~(1u << i); \
            _Pragma("unroll") for (int ff = 0; ff < 4; ++ff) acc[ff] = __builtin_amdgcn_mfma_f32_16x16x32_f16( \
                __builtin_bit_cast(f16x8, bf[i * 4 + ff]), __builtin_bit_cast(f16x8, af[i]), acc[ff], 0, 0, 0); } }
        issue8(af, ab, va);
        wait_frag<7>(af[0]); STEP(0) wait_frag<6>(af[1]); STEP(1) wait_frag<5>(af[2]); STEP(2) wait_frag<4>(af[3]); STEP(3)
        wait_frag<3>(af[4]); STEP(4) wait_frag<2>(af[5]); STEP(5) wait_frag<1>(af[6]); STEP(6) wait_frag<0>(af[7]); STEP(7)
        if (pend != 0u) {
            unsigned spins = 0;
            const unsigned t0 = now32();
            while (ok && pend != 0u) {
                load8(af, pend, ab, va);
                STEP(0) STEP(1) STEP(2) STEP(3) STEP(4) STEP(5) STEP(6) STEP(7)
                if (give_up(spins, t0, tmo, call)) ok = false;
            }
        }
        STAMP(2 + it);
        if (!ok) { float nn; asm volatile("v_mov_b32 %0, 0x7fc00000" : "=v"(nn)); acc[0][0] = nn; }
        float* pb = &part[(it & 1) * 8 * 16 * PPITCH];
#pragma unroll
        for (int i = 0; i < 4; ++i) {
            f32x4 q0 = {acc[0][i], acc[1][i], acc[2][i], acc[3][i]};
            *(f32x4*)(pb + pofs + i * 4) = q0;
        }
        asm volatile("s_waitcnt lgkmcnt(0)\n\ts_barrier" ::: "memory");
        if (it == 0) tout = lds_r32(L_TOUT);
        f32x2 t[8];
#pragma unroll
        for (int w = 0; w < 8; ++w) t[w] = *(const f32x2*)(pb + eofs + w * 16 * PPITCH);
        asm volatile("" : "+v"(t[0]), "+v"(t[1]), "+v"(t[2]), "+v"(t[3]), "+v"(t[4]), "+v"(t[5]), "+v"(t[6]), "+v"(t[7]));
        const f32x2 s = ((t[0] + t[1]) + (t[2] + t[3])) + ((t[4] + t[5]) + (t[6] + t[7]));
        u32x2 on = {gelu_bits(s[0] * (1.0f / WSCALE_F)), gelu_bits(s[1] * (1.0f / WSCALE_F))};
        asm volatile("" : "+v"(on));
        if (held) { wait_until(otime); store_wt2(oaddr, vo, o); STAMP(11); }
        o = on;
        oaddr = ob + (size_t)((item >> 8) & 0xffffu) * (NLOC * 4);
        const unsigned turn = (unsigned)((it == 0 ? (pos + ROT) & 31 : pos) * 8 + wv);
        const unsigned t1 = tout + turn * NS_OUT / 200u, t2 = tlate + turn * NS_LATE / 200u;
        otime = !remote ? tstart : (int)(t2 - t1) > 0 ? t2 : t1;
        held = true;
    }
    wait_until(otime);
    store_wt2(oaddr, vo, o);
    STAMP(7);
    asm volatile("s_waitcnt vmcnt(0)" ::: "memory");
}

extern "C" void kernel_launch(int rank, int nranks, ncclComm_t comm, hipStream_t stream,
                              void* const* d_in, const int* in_sizes, int n_in,
                              void* d_out, int out_size, const int* in_dtypes, int out_dtype,
                              void* const* d_out_all, void* const* d_ws_all, size_t ws_bytes) {
    if (nranks != NR || (unsigned)rank >= (unsigned)NR || ws_bytes < WS_NEED) { fprintf(stderr, "bad config\n"); return; }
    Args a;
    const int T = rank >> 2, idx = rank & 3;
    a.out = (float*)d_out;
    a.self = (char*)d_ws_all[rank];
    for (int v = 0; v < 8; ++v) {
        Wave& t = a.wave[v];
        const bool wrole = v >= 3 && v < 7;
        const int first = v == 0 ? 0 : v == 1 ? NX : v == 2 ? 1 : NX > 11 ? 11 : 12;
        const int rest = v < 3 ? 2 + 3 * v : 13;
        t.src = wrole ? (const char*)d_in[1] + (v - 3) * 256 : (const char*)d_in[0];
        t.kstep = wrole ? 32768u | 4u : 128u | 3u;
        t.step = wrole ? 1024u : 65536u;
        t.off0 = wrole ? 0u : (unsigned)first * 131072u;
        t.off2 = wrole ? 2048u : (unsigned)rest * 131072u;
        t.link = (char*)d_ws_all[v < 3 ? T * 4 + ((idx + v + 1) & 3) : wrole ? (T ^ 1) * 4 + v - 3 : rank];
    }
    for (int q = 0; q < 4; ++q) {
        a.outp[q] = (float*)d_out + 64 * q;
        a.outp[4 + q] = (float*)d_out_all[(T ^ 1) * 4 + q] + 64 * idx;
    }
    int key[64]; unsigned val[64]; int n = 0;
    for (int s = 0; s < 4; ++s)
        for (int i = 0; i < NX; ++i) {
            key[n] = s == idx ? 0 : i + 1;
            val[n] = (unsigned)(s * NX + i) | (unsigned)((T * 4 + s) * 256 + i * 16) << 8 | (unsigned)i << 24;
            const unsigned r = (unsigned)(((T ^ 1) * 4 + s) * 256 + i * 16);
            a.tiles[n++] = r | (i > 0 ? r - 16 : r) << 16;
        }
    for (int s = 0; s < 4; ++s)
        for (int i = 0; i < NY; ++i) {
            key[n] = i == 0 ? (WPOS < 64 ? 5 : 1) : 5 + i;
            val[n] = (unsigned)(4 * NX + s * NY + i) | (unsigned)(((T ^ 1) * 4 + s) * 256 + (NX + i) * 16) << 8 | (unsigned)(WPOS < 64 ? (i ? i : 128) : i > 1 ? i - 1 : 0) << 24;
            const unsigned r = (unsigned)((T * 4 + s) * 256 + (NX + i) * 16);
            a.tiles[n++] = r | (i > 0 ? r - 16 : r) << 16;
        }
    int pos = 0;
    for (int k = 0; k <= 32; ++k)
        for (int j = 0; j < 64; ++j)
            if (key[j] == k) a.items[pos++] = val[j];
    a.idx = idx;
    a.pad = 0;
    ag_gemm<<<NWG, NTHR, 0, stream>>>(a);
    HIP_CHECK(hipGetLastError());
}
```

```cpp
#include <hip/hip_runtime.h>
#include <rccl/rccl.h>
#include <cstdio>
#include <algorithm>

#define HIP_CHECK(expr) do { hipError_t _e = (expr); if (_e != hipSuccess) { \
    fprintf(stderr, "HIP error %s at %s:%d\n", hipGetErrorString(_e), __FILE__, __LINE__); \
    return; } } while (0)

typedef unsigned u32x4 __attribute__((ext_vector_type(4)));
typedef unsigned u32x2 __attribute__((ext_vector_type(2)));
typedef float f32x4 __attribute__((ext_vector_type(4)));
typedef float f32x2 __attribute__((ext_vector_type(2)));
typedef _Float16 f16x8 __attribute__((ext_vector_type(8)));
typedef _Float16 f16x2 __attribute__((ext_vector_type(2)));

#define NS_X 175
#define NS_OUT 173
#define NS_LATE 195
#define T_BASE 2
#define T_ADJ 0
#define LWOFF 24
#define PFOFF 48
#define LD0 10
#define LD_STEP 40
#define KL0 60
#define CLR_START 500
#define CLR_STEP 5
#define BASE_READ 250
#define MQ1 4
#define TSLP 12
#define BTR 235
#define BTO 250
#define BTN 60
#define EB0 150
#define MGB 20
#define MG -20
#define MGT -65
#define TAILK 576
#define ST1 6
#define AH0 30
#define AH1 70
#define TB_GAP 60
#define GUARD 55
#define GA 60
#define GB 10
#define GC 0
#define ROT 6
#define TAILT 47
#define PD 8
#define VDEPTH 3
#define VGAP 4

constexpr int NR = 8;
constexpr int NX = 11, NY = 16 - NX;
constexpr int WPOS = 64, YPOS = 320, OUT_POS = 576;
constexpr int NLOC = 256;
constexpr int NWG = 256, NTHR = 512;
constexpr int WATCH = 64 + 12 * 8 + 6;
constexpr int NRT = 64 - NY;
constexpr int NFIRST = NRT - 32;
constexpr int NSLOT = 5;
constexpr size_t OFF_CNT = 0;
constexpr size_t OFF_TMO = 32768;
constexpr size_t OFF_TS = 33024;
constexpr size_t OFF_BASE = 33280;
constexpr size_t OFF_T0 = 33536;
constexpr size_t OFF_DIAG = 65536;
constexpr size_t OFF_DATA = 1u << 20;
constexpr size_t A_BYTES = (64u + NY) * 65536u;
constexpr size_t SLOT_BYTES = A_BYTES + 8u * 262144u;
constexpr size_t WS_NEED = OFF_DATA + NSLOT * SLOT_BYTES;
constexpr unsigned DEADLINE_TICKS = 50000000u;
constexpr float WSCALE_F = 64.0f;
constexpr int PPITCH = 68;
constexpr unsigned HOLE = 0xffffffffu;
static_assert(TAILT - PD >= NFIRST, "");

struct Wave {
    const char* src;
    unsigned kstep;
    unsigned step;
    unsigned off0;
    int idx;
    char* link;
    char* self;
    char* out;
    char* pad[2];
};
struct Args {
    Wave wave[8];
    float* outp[8];
    unsigned work[392];
    unsigned preds[64];
    unsigned tiles[64];
};

__device__ __forceinline__ void store_wt(char* sbase, unsigned voff, u32x4 v) {
    asm volatile("global_store_dwordx4 %0, %1, %2 sc0 sc1\n\ts_nop 1" :: "v"(voff), "v"(v), "s"(sbase) : "memory");
}
__device__ __forceinline__ void store_wt2(char* sbase, unsigned voff, u32x2 v) {
    asm volatile("global_store_dwordx2 %0, %1, %2 sc0 sc1\n\ts_nop 1" :: "v"(voff), "v"(v), "s"(sbase) : "memory");
}
__device__ __forceinline__ unsigned pack2(float a, float b) {
    f16x2 h = {(_Float16)a, (_Float16)b};
    unsigned u = __builtin_bit_cast(unsigned, h);
    return u == 0u ? 0x80000000u : u;
}
__device__ __forceinline__ unsigned gelu_bits(float s) {
    const float k0 = -2.0f * 0.7978845608028654f * 1.4426950408889634f / WSCALE_F;
    const float k1 = k0 * 0.044715f / (WSCALE_F * WSCALE_F);
    const float e = __builtin_amdgcn_exp2f(s * __builtin_fmaf(s * s, k1, k0));
    const unsigned u = __builtin_bit_cast(unsigned, s * __builtin_amdgcn_rcpf(__builtin_fmaf(e, WSCALE_F, WSCALE_F)));
    return min(u, HOLE - 1u);
}
__device__ __forceinline__ unsigned long long now() { return __builtin_amdgcn_s_memrealtime(); }
__device__ __forceinline__ unsigned now32() { return (unsigned)__builtin_amdgcn_s_memrealtime(); }
__device__ __forceinline__ bool give_up(unsigned& spins, unsigned t0, unsigned* tmo, unsigned call) {
    if (__builtin_expect((++spins & 63u) != 0u, 1)) return false;
    unsigned t, u;
    asm volatile("v_mov_b32 %0, 0\n\tglobal_load_dword %0, %0, %1 sc1\n\ts_waitcnt vmcnt(0)" : "=&v"(t) : "s"(tmo) : "memory");
    if (__builtin_amdgcn_readfirstlane(t) == call || now32() - t0 > DEADLINE_TICKS) {
        asm volatile("v_mov_b32 %0, 0\n\tv_mov_b32 %1, %3\n\tglobal_store_dword %0, %1, %2 sc1\n\ts_nop 1"
                     : "=&v"(t), "=&v"(u) : "s"(tmo), "s"(call) : "memory");
        return true;
    }
    return false;
}
__device__ __forceinline__ void wait_until(unsigned t) {
    while ((int)(now32() - t) < 0) __builtin_amdgcn_s_sleep(1);
}
template <int NS> __device__ __forceinline__ unsigned pos_time(unsigned base, int pos) { return base + (unsigned)(pos * NS) / 100u; }

__device__ __forceinline__ void issue_ts(unsigned& v, const char* p, int lane) {
    asm volatile("global_load_dword %0, %1, %2 sc0 sc1" : "=&v"(v) : "v"((unsigned)(lane & 7) * 4u), "s"(p) : "memory");
}
__device__ __forceinline__ unsigned mean_base(unsigned v, unsigned tstart) {
    const unsigned d = v - tstart + 100u;
    unsigned sum = 0, n = 0;
#pragma unroll
    for (int i = 0; i < 8; ++i) {
        const unsigned e = __builtin_amdgcn_readlane(d, i);
        sum += e < 400u ? e : 0u;
        n += e < 400u ? 1u : 0u;
    }
    return n ? tstart - 100u + sum / n : tstart + 100u;
}
__device__ __forceinline__ unsigned earliest(unsigned v, unsigned tstart) {
    unsigned d = tstart - v;
    d = d < 100u ? d : 0u;
    unsigned m = 0;
#pragma unroll
    for (int i = 0; i < 8; ++i) m = max(m, (unsigned)__builtin_amdgcn_readlane(d, i));
    return tstart - m;
}
constexpr unsigned L_CNT = 16384, L_TREF = 16388, L_CALL = 16400;
__device__ __forceinline__ void put_frag(int blk, unsigned lw, f32x4 p, f32x4 q) {
    u32x2 a = {pack2(p[0], p[1]), pack2(p[2], p[3])}, c = {pack2(q[0], q[1]), pack2(q[2], q[3])};
    asm volatile("ds_write_b64 %0, %1\n\tds_write_b64 %0, %2 offset:512" :: "v"(lw + blk * 1024), "v"(a), "v"(c) : "memory");
}
__device__ __forceinline__ void get_frag(u32x4& h, int blk, unsigned lr) {
    asm volatile("ds_read_b128 %0, %1 offset:%2" : "=&v"(h) : "v"(lr), "n"(blk * 1024) : "memory");
}
__device__ __forceinline__ void lds_w32(unsigned ad, unsigned v) { asm volatile("ds_write_b32 %0, %1" :: "v"(ad), "v"(v) : "memory"); }
__device__ __forceinline__ unsigned lds_r32(unsigned ad) {
    unsigned v;
    asm volatile("ds_read_b32 %0, %1\n\ts_waitcnt lgkmcnt(0)" : "=&v"(v) : "v"(ad) : "memory");
    return __builtin_amdgcn_readfirstlane(v);
}
__device__ __forceinline__ void lds_add(unsigned ad, unsigned v) {
    unsigned long long e;
    asm volatile("s_waitcnt lgkmcnt(0)\n\ts_mov_b64 %0, exec\n\ts_mov_b64 exec, 1\n\tds_add_u32 %1, %2\n\ts_mov_b64 exec, %0"
                 : "=&s"(e) : "v"(ad), "v"(v) : "memory");
}
__device__ __forceinline__ void lds_wait(unsigned want, unsigned mask) {
    while ((lds_r32(L_CNT) & mask) != want) __builtin_amdgcn_s_sleep(1);
}
__device__ __forceinline__ void lds_atleast(unsigned n) {
    while ((lds_r32(L_CNT) & 0xffu) < n) __builtin_amdgcn_s_sleep(1);
}
__device__ __forceinline__ void issue2(f32x4* r, const char* sb, unsigned o) {
    asm volatile("global_load_dwordx4 %0, %2, %4\n\tglobal_load_dwordx4 %1, %3, %4"
                 : "=&v"(r[0]), "=&v"(r[1]) : "v"(o), "v"(o + 65536u), "s"(sb) : "memory");
}
constexpr int NORD = 13;
constexpr int ORDER[13] = {2, 3, 4, 12, 5, 13, 6, 14, 7, 15, 8, 9, 10};
constexpr int READY[16] = {0, 0, 1, 2, 3, 5, 7, 9, 11, 12, 13, 0, 4, 6, 8, 10};

#define LD1(i, off) "s_bitcmp1_b32 %[pend], " #i "\n\ts_cbranch_scc0 Lx" #i "_%=\n\tglobal_load_dwordx4 %" #i ", %[va], %[sb] offset:" #off " sc0 sc1\nLx" #i "_%=:\n\t"
__device__ __forceinline__ void load8(u32x4* r, unsigned pend, const char* sb, unsigned va) {
    asm volatile("s_nop 4\n\t"
                 LD1(0, -4096) LD1(1, -3072) LD1(2, -2048) LD1(3, -1024) LD1(4, 0) LD1(5, 1024) LD1(6, 2048) LD1(7, 3072)
                 "s_waitcnt vmcnt(0)"
                 : "+v"(r[0]), "+v"(r[1]), "+v"(r[2]), "+v"(r[3]), "+v"(r[4]), "+v"(r[5]), "+v"(r[6]), "+v"(r[7])
                 : [pend] "s"(pend), [va] "v"(va), [sb] "s"(sb)
                 : "memory", "scc");
}
__device__ __forceinline__ void issue8(u32x4* r, const char* sb, unsigned va) {
    asm volatile("s_nop 4\n\t"
                 "global_load_dwordx4 %0, %[va], %[sb] offset:-4096 sc0 sc1\n\t"
                 "global_load_dwordx4 %1, %[va], %[sb] offset:-3072 sc0 sc1\n\t"
                 "global_load_dwordx4 %2, %[va], %[sb] offset:-2048 sc0 sc1\n\t"
                 "global_load_dwordx4 %3, %[va], %[sb] offset:-1024 sc0 sc1\n\t"
                 "global_load_dwordx4 %4, %[va], %[sb] offset:0 sc0 sc1\n\t"
                 "global_load_dwordx4 %5, %[va], %[sb] offset:1024 sc0 sc1\n\t"
                 "global_load_dwordx4 %6, %[va], %[sb] offset:2048 sc0 sc1\n\t"
                 "global_load_dwordx4 %7, %[va], %[sb] offset:3072 sc0 sc1"
                 : "=&v"(r[0]), "=&v"(r[1]), "=&v"(r[2]), "=&v"(r[3]), "=&v"(r[4]), "=&v"(r[5]), "=&v"(r[6]), "=&v"(r[7])
                 : [va] "v"(va), [sb] "s"(sb)
                 : "memory");
}
__device__ __forceinline__ void issue_a(const char* sb, unsigned va) {
    asm volatile("s_nop 4\n\t"
                 "global_load_dwordx4 v[224:227], %0, %1 offset:-4096 sc0 sc1\n\t"
                 "global_load_dwordx4 v[228:231], %0, %1 offset:-3072 sc0 sc1\n\t"
                 "global_load_dwordx4 v[232:235], %0, %1 offset:-2048 sc0 sc1\n\t"
                 "global_load_dwordx4 v[236:239], %0, %1 offset:-1024 sc0 sc1\n\t"
                 "global_load_dwordx4 v[240:243], %0, %1 offset:0 sc0 sc1\n\t"
                 "global_load_dwordx4 v[244:247], %0, %1 offset:1024 sc0 sc1\n\t"
                 "global_load_dwordx4 v[248:251], %0, %1 offset:2048 sc0 sc1\n\t"
                 "global_load_dwordx4 v[252:255], %0, %1 offset:3072 sc0 sc1"
                 :: "v"(va), "s"(sb)
                 : "memory", "v224", "v225", "v226", "v227", "v228", "v229", "v230", "v231", "v232", "v233", "v234", "v235", "v236", "v237", "v238", "v239", "v240", "v241", "v242", "v243", "v244", "v245", "v246", "v247", "v248", "v249", "v250", "v251", "v252", "v253", "v254", "v255");
}
#define CHECK_A(n, r0, r1, r2, r3, miss) { unsigned tv_; \
    asm volatile("s_waitcnt vmcnt(" #n ")\n\tv_min_u32 %0, " r2 ", " r3 "\n\tv_min3_u32 %0, " r0 ", " r1 ", %0" : "=&v"(tv_) :: "memory"); \
    miss = __ballot(tv_ == 0u); asm volatile("" :: "s"(miss)); }
#define TAKE_A(reg, af) asm volatile("" : "={" reg "}"(af) :: "memory");
#define LOAD_A(reg, off, r0, r1, r2, r3, base, lanes) asm volatile("s_mov_b64 exec, %2\n\tglobal_load_dwordx4 " reg ", %0, %1 offset:" #off " sc0 sc1\n\ts_mov_b64 exec, -1" \
    :: "v"(vx), "s"(base), "s"(lanes) : "memory", r0, r1, r2, r3);
#define LOAD_0(base, lanes) LOAD_A("v[224:227]", -4096, "v224", "v225", "v226", "v227", base, lanes)
#define LOAD_1(base, lanes) LOAD_A("v[228:231]", -3072, "v228", "v229", "v230", "v231", base, lanes)
#define LOAD_2(base, lanes) LOAD_A("v[232:235]", -2048, "v232", "v233", "v234", "v235", base, lanes)
#define LOAD_3(base, lanes) LOAD_A("v[236:239]", -1024, "v236", "v237", "v238", "v239", base, lanes)
#define LOAD_4(base, lanes) LOAD_A("v[240:243]", 0, "v240", "v241", "v242", "v243", base, lanes)
#define LOAD_5(base, lanes) LOAD_A("v[244:247]", 1024, "v244", "v245", "v246", "v247", base, lanes)
#define LOAD_6(base, lanes) LOAD_A("v[248:251]", 2048, "v248", "v249", "v250", "v251", base, lanes)
#define LOAD_7(base, lanes) LOAD_A("v[252:255]", 3072, "v252", "v253", "v254", "v255", base, lanes)
#define CHECK_0(n, miss) CHECK_A(n, "v224", "v225", "v226", "v227", miss)
#define TAKE_0(af) TAKE_A("v[224:227]", af)
#define CHECK_1(n, miss) CHECK_A(n, "v228", "v229", "v230", "v231", miss)
#define TAKE_1(af) TAKE_A("v[228:231]", af)
#define CHECK_2(n, miss) CHECK_A(n, "v232", "v233", "v234", "v235", miss)
#define TAKE_2(af) TAKE_A("v[232:235]", af)
#define CHECK_3(n, miss) CHECK_A(n, "v236", "v237", "v238", "v239", miss)
#define TAKE_3(af) TAKE_A("v[236:239]", af)
#define CHECK_4(n, miss) CHECK_A(n, "v240", "v241", "v242", "v243", miss)
#define TAKE_4(af) TAKE_A("v[240:243]", af)
#define CHECK_5(n, miss) CHECK_A(n, "v244", "v245", "v246", "v247", miss)
#define TAKE_5(af) TAKE_A("v[244:247]", af)
#define CHECK_6(n, miss) CHECK_A(n, "v248", "v249", "v250", "v251", miss)
#define TAKE_6(af) TAKE_A("v[248:251]", af)
#define CHECK_7(n, miss) CHECK_A(n, "v252", "v253", "v254", "v255", miss)
#define TAKE_7(af) TAKE_A("v[252:255]", af)
#define CHECK_8(n, miss)
#define LOAD_9(base, lanes)
__device__ __forceinline__ void some_a(unsigned mask, const char* sb, unsigned va) {
    asm volatile("s_bitcmp1_b32 %2, 0\n\ts_cbranch_scc0 Ly0_%=\n\tglobal_load_dwordx4 v[224:227], %0, %1 offset:-4096 sc0 sc1\nLy0_%=:\n\t"
                 "s_bitcmp1_b32 %2, 1\n\ts_cbranch_scc0 Ly1_%=\n\tglobal_load_dwordx4 v[228:231], %0, %1 offset:-3072 sc0 sc1\nLy1_%=:\n\t"
                 "s_bitcmp1_b32 %2, 2\n\ts_cbranch_scc0 Ly2_%=\n\tglobal_load_dwordx4 v[232:235], %0, %1 offset:-2048 sc0 sc1\nLy2_%=:\n\t"
                 "s_bitcmp1_b32 %2, 3\n\ts_cbranch_scc0 Ly3_%=\n\tglobal_load_dwordx4 v[236:239], %0, %1 offset:-1024 sc0 sc1\nLy3_%=:\n\t"
                 "s_bitcmp1_b32 %2, 4\n\ts_cbranch_scc0 Ly4_%=\n\tglobal_load_dwordx4 v[240:243], %0, %1 offset:0 sc0 sc1\nLy4_%=:\n\t"
                 "s_bitcmp1_b32 %2, 5\n\ts_cbranch_scc0 Ly5_%=\n\tglobal_load_dwordx4 v[244:247], %0, %1 offset:1024 sc0 sc1\nLy5_%=:\n\t"
                 "s_bitcmp1_b32 %2, 6\n\ts_cbranch_scc0 Ly6_%=\n\tglobal_load_dwordx4 v[248:251], %0, %1 offset:2048 sc0 sc1\nLy6_%=:\n\t"
                 "s_bitcmp1_b32 %2, 7\n\ts_cbranch_scc0 Ly7_%=\n\tglobal_load_dwordx4 v[252:255], %0, %1 offset:3072 sc0 sc1\nLy7_%=:"
                 :: "v"(va), "s"(sb), "s"(mask)
                 : "memory", "scc", "v224", "v225", "v226", "v227", "v228", "v229", "v230", "v231", "v232", "v233", "v234", "v235", "v236", "v237", "v238", "v239", "v240", "v241", "v242", "v243", "v244", "v245", "v246", "v247", "v248", "v249", "v250", "v251", "v252", "v253", "v254", "v255");
}
template <int N> __device__ __forceinline__ void wait8(u32x4* r) {
    asm volatile("s_waitcnt vmcnt(%8)"
                 : "+v"(r[0]), "+v"(r[1]), "+v"(r[2]), "+v"(r[3]), "+v"(r[4]), "+v"(r[5]), "+v"(r[6]), "+v"(r[7]) : "n"(N) : "memory");
}
__device__ __forceinline__ unsigned min4(u32x4 v) { return min(min(v[0], v[1]), min(v[2], v[3])); }
__device__ __forceinline__ unsigned max4(u32x4 v) { return max(max(v[0], v[1]), max(v[2], v[3])); }
__device__ __forceinline__ bool arrived(u32x4 v) { return __ballot(min4(v) == 0u) == 0ull; }
__device__ __forceinline__ bool arrived8(const u32x4* r) {
    unsigned m = min4(r[0]);
#pragma unroll
    for (int i = 1; i < 8; ++i) m = min(m, min4(r[i]));
    return __ballot(m == 0u) == 0ull;
}

__device__ __forceinline__ void issue_tile(u32x4* v, const char* tl, unsigned to, unsigned t2) {
    asm volatile("global_load_dwordx4 %0, %4, %6 offset:-4096 sc0 sc1\n\tglobal_load_dwordx4 %1, %4, %6 sc0 sc1\n\t"
                 "global_load_dwordx4 %2, %5, %6 offset:-4096 sc0 sc1\n\tglobal_load_dwordx4 %3, %5, %6 sc0 sc1"
                 : "=&v"(v[0]), "=&v"(v[1]), "=&v"(v[2]), "=&v"(v[3]) : "v"(to), "v"(t2), "s"(tl) : "memory");
}
template <int SLP, unsigned EMPTY, int MISSING = 0>
__device__ __forceinline__ bool wait_words(const char* sb, unsigned vo, unsigned* tmo, unsigned call) {
    unsigned spins = 0;
    const unsigned t0 = now32();
    while (true) {
        unsigned v;
        asm volatile("global_load_dword %0, %1, %2 sc0 sc1\n\ts_waitcnt vmcnt(0)" : "=&v"(v) : "v"(vo), "s"(sb) : "memory");
        if (__builtin_popcountll(__ballot(v == EMPTY)) <= MISSING) return true;
        if (give_up(spins, t0, tmo, call)) return false;
        __builtin_amdgcn_s_sleep(SLP);
    }
}

#define FINISH() do { asm volatile("s_waitcnt vmcnt(0)" ::: "memory"); __builtin_amdgcn_endpgm(); } while (0)

__global__ __launch_bounds__(NTHR, 2) __attribute__((amdgpu_num_vgpr(216))) void ag_gemm(Args a) {
    __shared__ __attribute__((aligned(16))) float part[2 * 8 * 16 * PPITCH + 4];
    const int tid = threadIdx.x;
    const int lane = tid & 63;
    const int wv = __builtin_amdgcn_readfirstlane(tid >> 6);
    const int b = blockIdx.x;
    const Wave wt = a.wave[wv];
    char* self = wt.self;
    char* dst = wt.link;
    char* outb = wt.out;
    const int idx = wt.idx;
    asm volatile("" :: "s"(self), "s"(dst), "s"(wt.src), "s"(wt.kstep), "s"(wt.step), "s"(wt.off0), "s"(idx), "s"(outb));
    unsigned* cnt = (unsigned*)(self + OFF_CNT + b * 128 + wv * 4);
    unsigned* tmo = (unsigned*)(self + OFF_TMO);
    if (__builtin_expect(b < 64, 1)) {
        const int kf = b;
        const int kd = (b & 7) * 8 + (b >> 3);
        const size_t ko = (size_t)kd * 1024;
        unsigned prev;
        unsigned long long t64;
        f32x4 r[8];
        const unsigned sh = wt.kstep & 31u;
        const unsigned ol = (((unsigned)lane >> sh) << 13) + (((unsigned)lane & ((1u << sh) - 1u)) << 4) + (unsigned)kd * (wt.kstep & ~31u);
        {
            const unsigned o0 = ol + wt.off0;
            asm volatile("s_memrealtime %9\n\tv_mov_b32 %8, 0\n\ts_bitcmp1_b32 %[xm], %[wv]\n\ts_cbranch_scc0 La_%=\n\tglobal_load_dword %8, %8, %[cnt] sc1\n\t"
                         "global_load_dwordx4 %0, %[a0], %[lb]\n\tglobal_load_dwordx4 %1, %[a1], %[lb]\n\ts_barrier\n; TAG_X\n\ts_branch Lb_%=\n"
                         "La_%=:\n\ts_barrier\n\ts_cmp_lt_u32 %[wv], %[wl]\n; TAG_\n\ts_cbranch_scc0 Lb_%=\n; TAG_W\n\t"
                         "global_load_dwordx4 %0, %[a0], %[lb]\n\tglobal_load_dwordx4 %1, %[a0], %[lb] offset:1024\n\t"
                         "global_load_dwordx4 %2, %[a0], %[lb] offset:2048\n\tglobal_load_dwordx4 %3, %[a0], %[lb] offset:3072\n\t"
                         "global_load_dwordx4 %4, %[a1], %[lb]\n\tglobal_load_dwordx4 %5, %[a1], %[lb] offset:1024\n\t"
                         "global_load_dwordx4 %6, %[a1], %[lb] offset:2048\n\tglobal_load_dwordx4 %7, %[a1], %[lb] offset:3072\n"
                         "Lb_%=:\n\ts_waitcnt lgkmcnt(0)"
                         : "=&v"(r[0]), "=&v"(r[1]), "=&v"(r[2]), "=&v"(r[3]), "=&v"(r[4]), "=&v"(r[5]), "=&v"(r[6]), "=&v"(r[7]), "=&v"(prev), "=&s"(t64)
                         : [xm] "s"(0x07), [wv] "s"(wv), [wl] "n"(7), [lb] "s"(wt.src), [cnt] "s"(cnt), [a0] "v"(o0), [a1] "v"(o0 + wt.step)
                         : "memory", "scc");
        }
        const unsigned tstart = (unsigned)t64;
        const unsigned lw = (unsigned)(((((lane & 7) >> 1) * 8 + (lane >> 3)) * 16 + (lane & 1) * 8));
        const unsigned lr = (unsigned)(lane * 16);
        if (wv < 3) {
            asm volatile("; ROLE_X\n\ts_waitcnt vmcnt(0)" : "+v"(r[0]), "+v"(r[1]), "+v"(prev) :: "memory");
            put_frag(wv == 0 ? 0 : wv == 1 ? 11 : 1, lw, r[0], r[1]);
            if (wv == 0) lds_w32(L_CALL, prev + 1u);
            asm volatile("s_waitcnt lgkmcnt(0)\n\ts_barrier" ::: "memory");
            const unsigned tr = now32();
            u32x4 h;
            get_frag(h, 0, lr);
            const unsigned tb = tr + T_BASE;
            const unsigned call = __builtin_amdgcn_readfirstlane(prev) + 1u;
            char* sb = dst + OFF_DATA + (call % NSLOT) * SLOT_BYTES + ko + (size_t)(idx * NX) * 65536;
            asm volatile("s_waitcnt lgkmcnt(0)" : "+v"(h) :: "memory");
            wait_until(pos_time<NS_X>(tb, kf));
            store_wt(sb, lr, h);
            lds_wait(0x100u, 0x100u);
            const unsigned tref = lds_r32(L_TREF);
#pragma unroll
            for (int i = 1; i < NX; ++i) {
                if (i > 1) lds_atleast(READY[i]);
                get_frag(h, i, lr);
                asm volatile("s_waitcnt lgkmcnt(0)" : "+v"(h) :: "memory");
                wait_until(pos_time<NS_X>(tref, i * 64 + kf));
                store_wt(sb + (size_t)i * 65536, lr, h);
            }
            __hip_atomic_store(cnt, call, __ATOMIC_RELAXED, __HIP_MEMORY_SCOPE_AGENT);
            FINISH();
        } else if (wv < 7) {
            asm volatile("; ROLE_W");
            const int qd = wv - 3;
            const unsigned tile = a.tiles[b], preds = a.preds[b];
            char* tl = outb + (size_t)tile * (NLOC * 4) + qd * 256;
            const unsigned to = (unsigned)((lane >> 4) * (NLOC * 4) + (lane & 15) * 16 + 4096);
            const unsigned t2 = to + 8u * (NLOC * 4);
            const u32x4 hole = {HOLE, HOLE, HOLE, HOLE};
            asm volatile("s_barrier" ::: "memory");
            const unsigned tr = now32();
            u32x4 hh;
            get_frag(hh, 11, lr);
            const unsigned call = lds_r32(L_CALL);
            const unsigned tb = tr + T_BASE;
            asm volatile("" : "+v"(hh));
            const size_t so = OFF_DATA + (call % NSLOT) * SLOT_BYTES;
            char* sl = dst + so + ko;
            wait_until(pos_time<NS_X>(tb, kf));
            store_wt(sl + (size_t)(60 + idx) * 65536, lr, hh);
            lds_wait(0x100u, 0x100u);
            const unsigned tref = lds_r32(L_TREF);
            wait8<1>((u32x4*)r);
            u32x4 hw[4];
#pragma unroll
            for (int f = 0; f < 4; ++f) {
#pragma unroll
                for (int j = 0; j < 4; ++j) hw[f][j] = pack2(r[2 * j][f] * WSCALE_F, r[2 * j + 1][f] * WSCALE_F);
                asm volatile("" : "+v"(hw[f]));
            }
            const size_t bo = so + A_BYTES + ko;
#pragma unroll
            for (int f = 0; f < 4; ++f) {
                wait_until(pos_time<NS_X>(tref, WPOS + 64 * f + kf));
                if (f == 3) asm volatile("s_waitcnt vmcnt(1)" ::: "memory");
                store_wt(dst + bo + (size_t)(4 + idx) * 262144 + f * 65536, lr, hw[f]);
                if (f < 2) {
                    wait_until(pos_time<NS_X>(tref, WPOS + 64 * f + kf + LWOFF));
#pragma unroll
                    for (int g = 0; g < 2; ++g) store_wt(self + bo + (size_t)qd * 262144 + (2 * f + g) * 65536, lr, hw[2 * f + g]);
                }
                if (f < 2 && kf < NRT) {
                    wait_until(pos_time<NS_X>(tref, WPOS + 64 * f + kf + PFOFF));
                    store_wt(tl - 4096, f == 0 ? to : t2, hole);
                    store_wt(tl, f == 0 ? to : t2, hole);
                }
            }
#pragma unroll
            for (int i = 0; i < 4; ++i) {
                lds_atleast(READY[12 + i]);
                get_frag(hh, 12 + i, lr);
                asm volatile("s_waitcnt lgkmcnt(0)" : "+v"(hh) :: "memory");
                wait_until(pos_time<NS_X>(tref, YPOS + i * 64 + kf));
                store_wt(sl + (size_t)(44 + idx * 4 + i) * 65536, lr, hh);
            }
            if (kf >= NRT) FINISH();
            bool ok = wait_words<2, HOLE>(outb + (size_t)(((preds >> (7 * qd)) & 127u) * 16u + 15u) * (NLOC * 4) + qd * 256, 252, tmo, call);
            unsigned spins = 0;
            const unsigned t0 = now32();
            u32x4 v[VDEPTH][4];
#pragma unroll
            for (int d = 0; d < VDEPTH - 1; ++d) { issue_tile(v[d], tl, to, t2); __builtin_amdgcn_s_sleep(VGAP); }
            while (ok) {
#pragma unroll
                for (int d = 0; d < VDEPTH; ++d) {
                    issue_tile(v[(d + VDEPTH - 1) % VDEPTH], tl, to, t2);
                    asm volatile("s_waitcnt vmcnt(%4)" : "+v"(v[d][0]), "+v"(v[d][1]), "+v"(v[d][2]), "+v"(v[d][3]) : "n"(4 * (VDEPTH - 1)) : "memory");
                    if (__ballot(max(max(max4(v[d][0]), max4(v[d][1])), max(max4(v[d][2]), max4(v[d][3]))) == HOLE) == 0ull) __builtin_amdgcn_endpgm();
                    __builtin_amdgcn_s_sleep(VGAP);
                }
                if (give_up(spins, t0, tmo, call)) ok = false;
            }
            __builtin_amdgcn_endpgm();
        } else {
            asm volatile("; ROLE_C" ::: "memory");
            lds_w32(L_CNT, 0u);
            unsigned tsv;
            asm volatile("s_waitcnt lgkmcnt(0)\n\ts_barrier" ::: "memory");
            const unsigned tb = now32() + T_BASE;
            issue_ts(tsv, self + OFF_TS, lane);
            u32x4 h;
            get_frag(h, 0, lr);
            const unsigned call = lds_r32(L_CALL);
            char* sy = self + OFF_DATA + (call % NSLOT) * SLOT_BYTES + ko;
            char* sl = sy + (size_t)(idx * NX) * 65536;
            sy += (size_t)(64 - NX) * 65536;
            asm volatile("s_waitcnt vmcnt(0)" : "+v"(tsv), "+v"(h) :: "memory");
            const unsigned tref = earliest(tsv, tstart) + (tb - tstart);
            lds_w32(L_TREF, tref);
            lds_add(L_CNT, 0x100u);
            if (b < 8) __hip_atomic_store((unsigned*)(self + OFF_BASE) + b, tref, __ATOMIC_RELAXED, __HIP_MEMORY_SCOPE_SYSTEM);
            const unsigned tl = pos_time<NS_X>(tb, kf) + LD0;
            wait_until(tl);
            store_wt(sl, lr, h);
            get_frag(h, 1, lr);
            asm volatile("s_waitcnt lgkmcnt(0)" : "+v"(h) :: "memory");
            store_wt(sl + 65536, lr, h);
            f32x4 p[3][2];
            issue2(p[0], wt.src, ol + ORDER[0] * 131072u);
            wait_until(tl + LD_STEP);
            issue2(p[1], wt.src, ol + ORDER[1] * 131072u);
#pragma unroll
            for (int n = 0; n < NORD; ++n) {
                if (n + 2 < NORD) {
                    wait_until(tl + (n + 2) * LD_STEP);
                    issue2(p[(n + 2) % 3], wt.src, ol + ORDER[n + 2] * 131072u);
                }
                asm volatile("s_waitcnt vmcnt(%2)" : "+v"(p[n % 3][0]), "+v"(p[n % 3][1])
                             : "n"((n >= 2) + (n >= 1) + 2 * (n + 1 < NORD) + 2 * (n + 2 < NORD)) : "memory");
                put_frag(ORDER[n], lw, p[n % 3][0], p[n % 3][1]);
                get_frag(h, ORDER[n], lr);
                lds_add(L_CNT, 1u);
                asm volatile("" : "+v"(h));
                store_wt((ORDER[n] < NX ? sl : sy) + (size_t)ORDER[n] * 65536, lr, h);
            }
            get_frag(h, NX, lr);
            asm volatile("s_waitcnt lgkmcnt(0)" : "+v"(h) :: "memory");
            store_wt(sy + (size_t)NX * 65536, lr, h);
            char* cb = self + OFF_DATA + ((call + 2) % NSLOT) * SLOT_BYTES + (size_t)b * (SLOT_BYTES / 64);
            const u32x4 z = {0u, 0u, 0u, 0u};
#pragma unroll 1
            for (int i = 0; i < (int)(SLOT_BYTES / 64 / 1024); ++i) {
                wait_until(tstart + CLR_START + i * CLR_STEP);
                store_wt(cb + i * 1024, lr, z);
            }
            FINISH();
        }
    }
    const unsigned tstart = now32();
    if (b < 72 && wv == 1) __hip_atomic_store((unsigned*)(self + OFF_TS) + (b - 64), tstart, __ATOMIC_RELAXED, __HIP_MEMORY_SCOPE_SYSTEM);
    const bool remote = b < 192;
    const unsigned call = __builtin_amdgcn_readfirstlane(__hip_atomic_load(cnt, __ATOMIC_RELAXED, __HIP_MEMORY_SCOPE_AGENT)) + 1u;
    __hip_atomic_store(cnt, call, __ATOMIC_RELAXED, __HIP_MEMORY_SCOPE_AGENT);
    const int xc = b & 7, j = ((b - 64) >> 3) & 15;
    const int cg = (xc & 1) * 2 + (j & 1) + (remote ? 4 : 0);
    const int pos0 = (xc >> 1) + 4 * (j >> 1);
    const unsigned* wk = a.work + (remote ? ((cg & 3) * 32 + pos0) * 2 : 256 + pos0 * 8);
    unsigned item = wk[0], q1 = wk[1], q2 = wk[2], q3 = wk[3], q4 = wk[4];
    const char* slotp = self + OFF_DATA + (call % NSLOT) * SLOT_BYTES;
    char* ob = (char*)a.outp[cg];
    const unsigned va = lane * 16 + 4096;
    const unsigned vx = (unsigned)((((lane >> 3) & 1) * 32 + (lane >> 4) * 8 + (lane & 7)) * 16 + 4096);
    const unsigned vs = (unsigned)(lane & 7) * 4u;
    const char* sp = self + OFF_T0 + (call % NSLOT) * 128;
    bool ok = true;
    u32x4 bf[32];
    if (b == WATCH) {
        ok = wait_words<1, 0u, 64 - 8 * MQ1>(slotp + (wv < 4 ? (size_t)(wv * NX + 1) * 65536 : A_BYTES + (size_t)wv * 262144), ((lane & 7) * 8u + 7u) * 1024u + 1020u, tmo, call);
        __hip_atomic_store((unsigned*)sp + wv, (now32() - (wv == idx ? 63u : 127u) * NS_X / 100u) | 1u, __ATOMIC_RELAXED, __HIP_MEMORY_SCOPE_SYSTEM);
        __hip_atomic_store((unsigned*)(self + OFF_T0 + ((call + 2) % NSLOT) * 128) + wv, 0u, __ATOMIC_RELAXED, __HIP_MEMORY_SCOPE_SYSTEM);
    }
    unsigned tv;
    const unsigned wofs = (unsigned)(56 + wv) * NS_X / 100u;
#define KNOWN(w) (__builtin_amdgcn_readlane(tv, ((w) >> 20) & 7u) != 0u)
#define DUE(w) (__builtin_amdgcn_readlane(tv, ((w) >> 20) & 7u) + (((w) >> 23) & 127u) * (8u * NS_X) / 100u + wofs + (((w) >> 30) & 1u ? MGT : MG))
    const char* ab = slotp + (size_t)(item & 127u) * 65536 + (size_t)wv * 8192;
    unsigned tout;
    bool have;
    {
        const char* bb = slotp + A_BYTES + (size_t)cg * 262144 + (size_t)wv * 8192;
        unsigned spins = 0, tsv;
        const unsigned t0 = now32();
        unsigned tp = tstart + (remote ? BTR : BTO) + wofs;
        if (remote) {
            wait_until(tstart + EB0);
            while ((int)(now32() + 50u - tp) < 0) {
                asm volatile("global_load_dword %0, %1, %2 sc0 sc1\n\ts_waitcnt vmcnt(0)" : "=&v"(tv) : "v"(vs), "s"(sp) : "memory");
                const unsigned e = __builtin_amdgcn_readlane(tv, cg);
                if (e) { if ((int)(e + 64u * NS_X / 100u + wofs + MGB - tp) < 0) tp = e + 64u * NS_X / 100u + wofs + MGB; break; }
                __builtin_amdgcn_s_sleep(TSLP);
            }
        }
#pragma unroll
        for (int g = 0; g < 4; ++g) {
            wait_until(tp);
            if (g == 3) {
                asm volatile("global_load_dword %0, %1, %2 sc0 sc1" : "=&v"(tv) : "v"(vs), "s"(sp) : "memory");
                wait_until(tstart + BASE_READ);
                issue_ts(tsv, self + OFF_BASE, lane);
            }
            tp += remote || g == 1 ? 64u * NS_X / 100u : 0u;
            while (ok) {
                unsigned v;
                asm volatile("global_load_dword %0, %1, %2 sc0 sc1\n\ts_waitcnt vmcnt(0)" : "=&v"(v) : "v"(7u * 1024u + 1020u), "s"(bb + g * 65536) : "memory");
                if (__ballot(v == 0u) == 0ull) break;
                if (give_up(spins, t0, tmo, call)) ok = false;
                __builtin_amdgcn_s_sleep(1);
                if (remote || g == 1) tp = now32() + BTN;
            }
            if (g == 3) asm volatile("s_waitcnt vmcnt(0)" : "+v"(tv), "+v"(tsv) :: "memory");
            issue8(&bf[g * 8], bb + g * 65536, va);
        }
        tout = pos_time<NS_X>(mean_base(tsv, tstart) + T_ADJ, OUT_POS);
        have = KNOWN(item) && (int)(now32() - DUE(item)) >= 0;
        issue_a(have ? ab : bb, vx);
        unsigned pend = 0;
        wait8<32>(&bf[0]);
        if (!arrived8(&bf[0])) pend |= 1u;
        wait8<24>(&bf[8]);
        if (!arrived8(&bf[8])) pend |= 2u;
        wait8<16>(&bf[16]);
        if (!arrived8(&bf[16])) pend |= 4u;
        wait8<8>(&bf[24]);
        if (!arrived8(&bf[24])) pend |= 8u;
        if (__builtin_expect(pend != 0u, 0)) {
            while (ok && pend != 0u) {
#pragma unroll
                for (int g = 0; g < 4; ++g)
                    if ((pend >> g) & 1u) {
                        load8(&bf[g * 8], 255u, bb + g * 65536, va);
                        if (arrived8(&bf[g * 8])) pend &= ~(1u << g);
                    }
                if (give_up(spins, t0, tmo, call)) ok = false;
            }
        }
    }
    unsigned tnow = now32();
    const unsigned tlate = tnow + TB_GAP;
    const int pofs = (wv * 16 + (lane & 15)) * PPITCH + (lane >> 4) * 16;
    const int erow = wv * 2 + (lane >> 5);
    const int eofs = erow * PPITCH + (lane & 31) * 2;
    const unsigned vo = (unsigned)(erow * NLOC * 4 + (lane & 31) * 8);
    u32x2 o = {0u, 0u};
    char* oaddr = ob;
    unsigned otime = 0;
    bool held = false;
#pragma unroll 1
    for (int it = 0; ; ++it) {
        const bool last = item >> 31;
        bool ahead = !last && KNOWN(q1) && (int)(tnow + AH0 - DUE(q1)) >= 0;
        const char* nb = slotp + (size_t)(q1 & 127u) * 65536 + (size_t)wv * 8192;
        const char* abn = ahead ? nb : ab;
        unsigned long long more = ahead ? ~0ull : 1ull;
        const unsigned turn = ((item >> 14) & 63u) * 8u + wv;
        const unsigned t1 = tout + turn * NS_OUT / 200u, t2 = tlate + turn * NS_LATE / 200u;
        const unsigned ntime = !remote ? tstart : (int)(t2 - t1) > 0 ? t2 : t1;
        char* naddr = ob + (size_t)((item >> 7) & 127u) * (16 * NLOC * 4);
        f32x4 acc[4] = {};
        unsigned pend = 255u;
#define MM(ff, i) acc[ff] = __builtin_amdgcn_mfma_f32_16x16x32_f16(__builtin_bit_cast(f16x8, bf[ff * 8 + i]), __builtin_bit_cast(f16x8, af), acc[ff], 0, 0, 0);
#define FENCE __builtin_amdgcn_sched_barrier(0);
#define STEP(i, h, j) if (__builtin_expect(miss == 0ull, 1)) { u32x4 af; TAKE_##i(af) \
                MM(0, i) FENCE LOAD_##h(abn, more) FENCE MM(1, i) FENCE CHECK_##j(7, miss) FENCE MM(2, i) pend = (254u << i) & 255u; MM(3, i) FENCE
#define LATE(i) if ((pend >> i) & 1u) { u32x4 af; TAKE_##i(af) if (arrived(af)) { pend &= ~(1u << i); MM(0, i) MM(1, i) MM(2, i) MM(3, i) } }
        unsigned spins = 0, t0 = tnow;
#pragma unroll 1
        while (true) {
            if (have) {
            const bool due = held && (int)(tnow + GA - otime) >= 0;
            if (due) { if ((int)(tnow - otime) < 0) wait_until(otime); held = false; }
            asm volatile("s_cmp_eq_u32 %[due], 0\n\ts_cbranch_scc1 Lj_%=\n\tglobal_store_dwordx2 %[vo], %[o], %[oa] sc0 sc1\n\ts_branch Lk_%=\n"
                         "Lj_%=:\n\tglobal_load_dword v220, %[vs], %[sp] sc0 sc1\nLk_%=:\n\ts_nop 1"
                         :: [due] "s"(__builtin_amdgcn_readfirstlane((int)due)), [vo] "v"(vo), [o] "v"(o), [oa] "s"(oaddr), [vs] "v"(vs), [sp] "s"(sp) : "memory", "scc", "v220");
            unsigned long long miss;
            CHECK_0(8, miss)
            STEP(0, 9, 1) STEP(1, 0, 2) STEP(2, 1, 3) STEP(3, 2, 4) STEP(4, 3, 5) STEP(5, 4, 6) STEP(6, 5, 7) STEP(7, 6, 8) LOAD_7(abn, more) }}}}}}}}
            if (__builtin_expect(pend == 0u, 1)) break;
            }
            if (!have) {
                while (ok && !KNOWN(item)) {
                    const unsigned tk = now32() + (spins ? TSLP * 3u : 0u);
                    if (held && (int)(tk + GUARD - otime) >= 0) { wait_until(otime); store_wt2(oaddr, vo, o); held = false; }
                    wait_until(tk);
                    asm volatile("global_load_dword %0, %1, %2 sc0 sc1\n\ts_waitcnt vmcnt(0)" : "=&v"(tv) : "v"(vs), "s"(sp) : "memory");
                    if (give_up(spins, t0, tmo, call)) ok = false;
                }
                const unsigned td = ok ? DUE(item) : tnow;
                if (held && (remote ? (int)(td - otime) >= 0 : (int)(td - now32()) > 50)) { wait_until(otime); store_wt2(oaddr, vo, o); held = false; }
                if ((item >> 30) & 1u) {
                    wait_until(td - (8u - ST1) * 8u * NS_X / 100u);
                    some_a((1u << ST1) - 1u, ab, vx);
                    wait_until(td);
                    some_a(255u << ST1, ab, vx);
                } else {
                    wait_until(td);
                    issue_a(ab, vx);
                }
                tnow = now32();
                have = true;
                ahead = !last && KNOWN(q1) && (int)(tnow + AH1 - DUE(q1)) >= 0;
                abn = ahead ? nb : ab;
                more = ahead ? ~0ull : 1ull;
                continue;
            }
            const unsigned rest = pend | pend >> 1;
            while (ok && pend != 0u) {
                if (held && (int)(now32() + GUARD - otime) >= 0) { wait_until(otime); store_wt2(oaddr, vo, o); held = false; }
                some_a(pend, ab, vx);
                asm volatile("s_waitcnt vmcnt(0)" ::: "memory");
                LATE(0) LATE(1) LATE(2) LATE(3) LATE(4) LATE(5) LATE(6) LATE(7)
                if (give_up(spins, t0, tmo, call)) ok = false;
                __builtin_amdgcn_s_sleep(1);
            }
            if (ahead) some_a(rest, abn, vx);
            break;
        }
        have = ahead;
        if (!ok) { float nn; asm volatile("v_mov_b32 %0, 0x7fc00000" : "=v"(nn)); acc[0][0] = nn; }
        float* pb = &part[(it & 1) * 8 * 16 * PPITCH];
#pragma unroll
        for (int i = 0; i < 4; ++i) {
            f32x4 q0 = {acc[0][i], acc[1][i], acc[2][i], acc[3][i]};
            *(f32x4*)(pb + pofs + i * 4) = q0;
        }
        if (GB && held && (int)(now32() + GB + (7 - wv) * GC - otime) >= 0) { wait_until(otime); store_wt2(oaddr, vo, o); held = false; }
        f32x2 t[8];
        unsigned long long t64;
        asm volatile("s_waitcnt lgkmcnt(0)\n\ts_barrier\n\t"
                     "ds_read_b64 %0, %9\n\tds_read_b64 %1, %9 offset:%10\n\tds_read_b64 %2, %9 offset:%10*2\n\tds_read_b64 %3, %9 offset:%10*3\n\t"
                     "ds_read_b64 %4, %9 offset:%10*4\n\tds_read_b64 %5, %9 offset:%10*5\n\tds_read_b64 %6, %9 offset:%10*6\n\tds_read_b64 %7, %9 offset:%10*7\n\t"
                     "s_memrealtime %8\n\ts_waitcnt lgkmcnt(0)"
                     : "=&v"(t[0]), "=&v"(t[1]), "=&v"(t[2]), "=&v"(t[3]), "=&v"(t[4]), "=&v"(t[5]), "=&v"(t[6]), "=&v"(t[7]), "=&s"(t64)
                     : "v"((__attribute__((address_space(3))) float*)(pb + eofs)), "n"(16 * PPITCH * 4) : "memory");
        tnow = (unsigned)t64;
        const f32x2 s = ((t[0] + t[1]) + (t[2] + t[3])) + ((t[4] + t[5]) + (t[6] + t[7]));
        u32x2 on = {gelu_bits(s[0]), gelu_bits(s[1])};
        asm volatile("" : "+v"(on));
        if (held) { wait_until(otime); store_wt2(oaddr, vo, o); }
        o = on;
        oaddr = naddr;
        otime = ntime;
        held = true;
        if (last) break;
        item = q1; q1 = q2; q2 = q3; q3 = q4;
        ab = nb;
    }
    if ((int)(tnow - otime) < 0) wait_until(otime);
    store_wt2(oaddr, vo, o);
    asm volatile("s_waitcnt vmcnt(0)" ::: "memory");
}

static int row_blocks(int T, int idx, int l, unsigned* list) {
    int key[64], src[64], n = 0, pos = 0;
    unsigned val[64];
    const unsigned me = T * 1024, other = (T ^ 1) * 1024;
    for (int s = 0; s < 4; ++s)
        for (int i = 0; i < NX; ++i) {
            key[n] = s == idx ? 0 : 64 * (i + 1);
            src[n] = -1;
            val[n++] = (unsigned)(s * NX + i) | (me + s * 256 + i * 16) / 16 << 7 | (unsigned)s << 20 | (s == idx ? 0u : 8u * i << 23 | (64 * i >= TAILK ? 1u << 30 : 0u));
        }
    for (int s = 0; s < 4; ++s)
        for (int i = 0; i < 4; ++i) {
            key[n] = YPOS + 64 * (i + 1);
            src[n] = s;
            val[n++] = (unsigned)(44 + s * 4 + i) | (other + s * 256 + (12 + i) * 16) / 16 << 7 | (4u + s) << 20 | (YPOS / 8 + 8u * i) << 23;
        }
    for (int s = 0; s < 4; ++s) {
        key[n] = 64;
        src[n] = s;
        val[n++] = (unsigned)(60 + s) | (other + s * 256 + 176) / 16 << 7 | (4u + s) << 20;
    }
    for (int i = 0; i < 16; ++i) {
        const int blk = i < 2 ? i : i < 15 ? ORDER[i - 2] : NX;
        const unsigned k8 = i < 2 ? 0u : ((unsigned)(std::min(i, 14) - 2) * LD_STEP + KL0) * 100u / (8u * NS_X) + 1u;
        if (blk < NX) list[pos++] = val[idx * NX + blk] | k8 << 23;
        else if (l == 4) list[pos++] = (unsigned)(64 + blk - NX) | (me + idx * 256 + blk * 16) / 16 << 7 | (unsigned)idx << 20 | k8 << 23;
    }
    for (int k = 64; k <= 64 * NX; k += 64)
        for (int j = 0; j < 64; ++j)
            if (key[j] == k && src[j] != l) list[pos++] = val[j];
    return pos;
}

extern "C" void kernel_launch(int rank, int nranks, ncclComm_t comm, hipStream_t stream,
                              void* const* d_in, const int* in_sizes, int n_in,
                              void* d_out, int out_size, const int* in_dtypes, int out_dtype,
                              void* const* d_out_all, void* const* d_ws_all, size_t ws_bytes) {
    if (nranks != NR || (unsigned)rank >= (unsigned)NR || ws_bytes < WS_NEED) { fprintf(stderr, "bad config\n"); return; }
    Args a = {};
    const int T = rank >> 2, idx = rank & 3;
    for (int v = 0; v < 8; ++v) {
        Wave& t = a.wave[v];
        const bool wrole = v >= 3 && v < 7;
        t.src = wrole ? (const char*)d_in[1] + (v - 3) * 256 : (const char*)d_in[0];
        t.kstep = wrole ? 32768u | 4u : 128u | 3u;
        t.step = wrole ? 4096u : 65536u;
        t.off0 = (v == 1 ? 11u : v == 2 ? 1u : 0u) * 131072u;
        t.idx = idx;
        t.self = (char*)d_ws_all[rank];
        t.out = (char*)d_out;
        t.link = (char*)d_ws_all[v < 3 ? T * 4 + ((idx + v + 1) & 3) : wrole ? (T ^ 1) * 4 + v - 3 : rank];
    }
    for (int q = 0; q < 4; ++q) {
        a.outp[q] = (float*)d_out + 64 * q;
        a.outp[4 + q] = (float*)d_out_all[(T ^ 1) * 4 + q] + 64 * idx;
    }
    unsigned list[72];
    for (int l = 0; l < 5; ++l) {
        const int pos = row_blocks(T, idx, l, list);
        if (l == 4) {
            for (int p = 0; p < pos; ++p) a.work[256 + (p & 15) * 8 + (p >> 4)] = list[p] | (p + 16 >= pos ? 1u << 31 : 0u);
        } else {
            for (int p = 0; p < NFIRST; ++p) a.work[(l * 32 + p) * 2] = list[p] | (unsigned)((p + ROT) % NFIRST) << 14;
            for (int p = 0; p < 32; ++p) a.work[(l * 32 + p) * 2 + (p < NFIRST)] = list[NFIRST + p] | (unsigned)(NFIRST + p) << 14 | 1u << 31;
        }
    }
    int nt = 0;
    for (int s = 0; s < 4; ++s)
        for (int i = 0; i < NX; ++i) a.tiles[nt++] = (T ^ 1) * 1024 + s * 256 + i * 16;
    for (int s = 0; s < 4; ++s)
        for (int i = 12; i < 16 && s != idx; ++i) a.tiles[nt++] = T * 1024 + s * 256 + i * 16;
    for (int s = 0; s < 4; ++s)
        if (s != idx) a.tiles[nt++] = T * 1024 + s * 256 + NX * 16;
    for (int q = 0; q < 4; ++q) {
        row_blocks(T ^ 1, q, idx, list);
        for (int p = 0; p < NRT; ++p)
            for (int t = 0; t < NRT; ++t)
                if (a.tiles[t] == ((list[p] >> 7) & 127u) * 16u) a.preds[t] |= ((list[p < TAILT ? p : p - PD] >> 7) & 127u) << (7 * q);
    }
    ag_gemm<<<NWG, NTHR, 0, stream>>>(a);
    HIP_CHECK(hipGetLastError());
}
```
